```python
import numpy as np

kernel_name = "gpu_a2a_gemm_m2048_k2048_n2048_f32_relu"


ABSMAX_RTOL = 0.001

COMPUTE_DTYPE = "float32"
EPILOGUE = "relu"
IN_QUANT = None
OUT_QUANT = None
IN_DTYPES = {"x": "float32", "w_mat": "float32"}
WORLD_SIZE = 8

def setup_inputs():

    x = np.random.randn(2048, 2048).astype(np.float32)
    w_mat = (np.random.randn(2048, 2048) / np.sqrt(2048)).astype(np.float32)
    return {"x": x, "w_mat": w_mat}

def _epilogue(y):
    return np.maximum(y, 0.0).astype(np.float32)

def reference(x, w_mat):
    y = (x @ w_mat).astype(np.float32)
    return _epilogue(y)
```

```cpp
#include <hip/hip_runtime.h>
#include <rccl/rccl.h>
#include <cstdio>
#include <cstdint>

#define HIP_CHECK(expr) do { hipError_t _e = (expr); if (_e != hipSuccess) { \
    fprintf(stderr, "HIP error %s at %s:%d\n", hipGetErrorString(_e), __FILE__, __LINE__); \
    return; } } while (0)

typedef _Float16 h2 __attribute__((ext_vector_type(2)));
typedef _Float16 h8 __attribute__((ext_vector_type(8)));
typedef float f2 __attribute__((ext_vector_type(2)));
typedef float f4 __attribute__((ext_vector_type(4)));
typedef unsigned u2 __attribute__((ext_vector_type(2)));
typedef unsigned u4 __attribute__((ext_vector_type(4)));
typedef __attribute__((address_space(1))) unsigned gu32;

constexpr int NR = 8;
constexpr int KD = 2048, ND = 2048;
constexpr int MS = 256;
constexpr int KS = 256;
constexpr int NWG = 256;
constexpr int NTHREADS = 576;

constexpr unsigned FLAG_OFF = 0;
constexpr unsigned CNT_OFF = 16384;
constexpr unsigned TMO_OFF = 65536;
constexpr unsigned PROF_OFF = 131072;
constexpr unsigned XG_OFF = 1u << 20;
constexpr unsigned XG_BYTES = 1u << 20;
constexpr unsigned WS_NEEDED = XG_OFF + 2 * XG_BYTES;
constexpr unsigned long long DEADLINE_TICKS = 50000000ull;

struct Args {
  const float* x;
  const float* w;
  float* out;
  char* myws;
  char* wsrot[NR];
  int rank;
  int pad;
};

#define STAMP(i) do {} while (0)

__device__ __forceinline__ unsigned cvt2(float a, float b) {
  f2 v = {a, b};
  return __builtin_bit_cast(unsigned, __builtin_convertvector(v, h2));
}

__global__ __launch_bounds__(NTHREADS, 3) void a2a_gemm_relu(Args a) {
  __shared__ __attribute__((aligned(16))) char red[8 * 8192];
  const int tid = threadIdx.x;
  const int lane = tid & 63;
  const int wave = __builtin_amdgcn_readfirstlane(tid >> 6);
  const unsigned b = blockIdx.x;
  const unsigned rank = (unsigned)a.rank;
  gu32* cnt = (gu32*)(a.myws + CNT_OFF + b * 128);
  const int c16 = lane & 15, q = lane >> 4;

  if (wave == 8) {
    const unsigned d = b & 7, u = b >> 3, m = u & 3, s = u >> 2;
    const unsigned r = (rank + 1 + d) & 7;
    const float* src = a.x + (size_t)(MS * r + 64 * m + c16) * KS + 32 * s + 8 * q;
    f4 v[4][2];
#pragma unroll
    for (int mt = 0; mt < 4; ++mt) {
      v[mt][0] = *(const f4*)(src + mt * 16 * KS);
      v[mt][1] = *(const f4*)(src + mt * 16 * KS + 4);
    }
    const unsigned call = __hip_atomic_load(cnt, __ATOMIC_RELAXED, __HIP_MEMORY_SCOPE_AGENT) + 1;
    char* dws = a.wsrot[d];
    char* dst = dws + XG_OFF + (call & 1) * XG_BYTES + ((rank * 4 + m) * 8 + s) * 4096 + lane * 16;
    u4 h[4];
#pragma unroll
    for (int mt = 0; mt < 4; ++mt) {
      h[mt].x = cvt2(v[mt][0].x, v[mt][0].y);
      h[mt].y = cvt2(v[mt][0].z, v[mt][0].w);
      h[mt].z = cvt2(v[mt][1].x, v[mt][1].y);
      h[mt].w = cvt2(v[mt][1].z, v[mt][1].w);
    }
    STAMP(1);
    asm volatile("global_store_dwordx4 %0, %1, off sc0 sc1\n\t"
                 "global_store_dwordx4 %0, %2, off offset:1024 sc0 sc1\n\t"
                 "global_store_dwordx4 %0, %3, off offset:2048 sc0 sc1\n\t"
                 "global_store_dwordx4 %0, %4, off offset:3072 sc0 sc1\n\t"
                 "s_waitcnt vmcnt(0)"
                 :: "v"(dst), "v"(h[0]), "v"(h[1]), "v"(h[2]), "v"(h[3]) : "memory");
    if (lane == 0)
      __hip_atomic_store((gu32*)(dws + FLAG_OFF + (rank * 4 + m) * 128 + s * 4), call, __ATOMIC_RELAXED,
                         __HIP_MEMORY_SCOPE_SYSTEM);
    STAMP(2);
    return;
  }

  const unsigned p = (unsigned)wave;
  const unsigned xcd = b & 7, j = b >> 3, m = j & 3, ns = j >> 2;
  const unsigned n0 = 256 * xcd + 32 * ns;
  const unsigned call = __hip_atomic_load(cnt, __ATOMIC_RELAXED, __HIP_MEMORY_SCOPE_AGENT) + 1;

  h8 wf[8][2];
  {
    __amdgpu_buffer_rsrc_t wr = __builtin_amdgcn_make_buffer_rsrc((void*)a.w, 0, KD * ND * 4, 0x00020000);
    const int voff = (8 * q * ND + 2 * c16) * 4;
    const int sbase = (KS * p * ND + n0) * 4;
#pragma unroll
    for (int s = 0; s < 8; ++s) {
      f2 raw[8];
#pragma unroll
      for (int i = 0; i < 8; ++i)
        raw[i] = __builtin_bit_cast(f2, __builtin_amdgcn_raw_buffer_load_b64(wr, voff, sbase + (32 * s + i) * ND * 4, 0));
      u4 a0, a1;
#pragma unroll
      for (int i = 0; i < 4; ++i) {
        a0[i] = cvt2(raw[2 * i].x, raw[2 * i + 1].x);
        a1[i] = cvt2(raw[2 * i].y, raw[2 * i + 1].y);
      }
      wf[s][0] = __builtin_bit_cast(h8, a0);
      wf[s][1] = __builtin_bit_cast(h8, a1);
    }
  }
#pragma unroll
  for (int s = 0; s < 8; ++s) { asm volatile("" : "+v"(wf[s][0]), "+v"(wf[s][1])); }
  STAMP(1);

  bool timed_out = false;
  {
    gu32* fl = (gu32*)(a.myws + FLAG_OFF + (p * 4 + m) * 128) + (lane & 7);
    gu32* tmo = (gu32*)(a.myws + TMO_OFF);
    const unsigned long long t0 = __builtin_amdgcn_s_memrealtime();
    for (;;) {
      const unsigned f = __hip_atomic_load(fl, __ATOMIC_RELAXED, __HIP_MEMORY_SCOPE_SYSTEM);
      if (__all((int)(f - call) >= 0)) break;
      if (__hip_atomic_load(tmo, __ATOMIC_RELAXED, __HIP_MEMORY_SCOPE_AGENT) == call ||
          __builtin_amdgcn_s_memrealtime() - t0 > DEADLINE_TICKS) {
        __hip_atomic_store(tmo, call, __ATOMIC_RELAXED, __HIP_MEMORY_SCOPE_AGENT);
        timed_out = true;
        break;
      }
      __builtin_amdgcn_s_sleep(1);
    }
    asm volatile("" ::: "memory");
  }
  STAMP(2);

  f4 acc[4][2];
#pragma unroll
  for (int mt = 0; mt < 4; ++mt) { acc[mt][0] = (f4)0.f; acc[mt][1] = (f4)0.f; }
  {
    char* xg = a.myws + XG_OFF + (call & 1) * XG_BYTES + (p * 4 + m) * 32768;
    __amdgpu_buffer_rsrc_t xr = __builtin_amdgcn_make_buffer_rsrc((void*)xg, 0, 32768, 0x00020000);
    const int voff = lane * 16;
    u4 xf[4][4];
#pragma unroll
    for (int s = 0; s < 4; ++s)
#pragma unroll
      for (int mt = 0; mt < 4; ++mt)
        xf[s][mt] = __builtin_amdgcn_raw_buffer_load_b128(xr, voff + mt * 1024, s * 4096, 17);
    __builtin_amdgcn_sched_barrier(0);
#pragma unroll
    for (int s = 0; s < 8; ++s) {
#pragma unroll
      for (int mt = 0; mt < 4; ++mt) {
        acc[mt][0] = __builtin_amdgcn_mfma_f32_16x16x32_f16(wf[s][0], __builtin_bit_cast(h8, xf[s & 3][mt]), acc[mt][0], 0, 0, 0);
        acc[mt][1] = __builtin_amdgcn_mfma_f32_16x16x32_f16(wf[s][1], __builtin_bit_cast(h8, xf[s & 3][mt]), acc[mt][1], 0, 0, 0);
        if (s < 4) xf[s][mt] = __builtin_amdgcn_raw_buffer_load_b128(xr, voff + mt * 1024, (s + 4) * 4096, 17);
        __builtin_amdgcn_sched_barrier(0);
      }
    }
  }
  if (timed_out) {
#pragma unroll
    for (int mt = 0; mt < 4; ++mt) { acc[mt][0] = (f4)__builtin_nanf(""); acc[mt][1] = (f4)__builtin_nanf(""); }
  }

#pragma unroll
  for (int mt = 0; mt < 4; ++mt) {
    const int row = 16 * mt + c16;
    f4 lo = {acc[mt][0][0], acc[mt][1][0], acc[mt][0][1], acc[mt][1][1]};
    f4 hi = {acc[mt][0][2], acc[mt][1][2], acc[mt][0][3], acc[mt][1][3]};
    *(f4*)(red + p * 8192 + row * 128 + (((2 * q) ^ (row & 7)) * 16)) = lo;
    *(f4*)(red + p * 8192 + row * 128 + (((2 * q + 1) ^ (row & 7)) * 16)) = hi;
  }
  __syncthreads();
  {
    const int row = 8 * p + (lane >> 3), seg = lane & 7;
    const char* rp = red + row * 128 + ((seg ^ (row & 7)) * 16);
    f4 sum = *(const f4*)rp;
#pragma unroll
    for (int w = 1; w < 8; ++w) sum += *(const f4*)(rp + w * 8192);
    f4 o;
#pragma unroll
    for (int i = 0; i < 4; ++i) o[i] = __builtin_elementwise_maximum(sum[i], 0.0f);
    *(f4*)(a.out + (size_t)(64 * m + row) * ND + n0 + 4 * seg) = o;
  }
  if (tid == 0) __hip_atomic_store(cnt, call, __ATOMIC_RELAXED, __HIP_MEMORY_SCOPE_AGENT);
  STAMP(4);
}

extern "C" void kernel_launch(int rank, int nranks, ncclComm_t comm, hipStream_t stream,
                              void* const* d_in, const int* in_sizes, int n_in,
                              void* d_out, int out_size,
                              const int* in_dtypes, int out_dtype,
                              void* const* d_out_all,
                              void* const* d_ws_all, size_t ws_bytes) {
  if (nranks != NR || (unsigned)rank >= (unsigned)NR || n_in < 2 || ws_bytes < WS_NEEDED ||
      in_sizes[0] != KD * KS || in_sizes[1] != KD * ND || out_size != MS * ND) {
    fprintf(stderr, "kernel_launch: unexpected shapes\n");
    return;
  }
  Args a;
  a.x = (const float*)d_in[0];
  a.w = (const float*)d_in[1];
  a.out = (float*)d_out;
  a.myws = (char*)d_ws_all[rank];
  for (int i = 0; i < NR; ++i) a.wsrot[i] = (char*)d_ws_all[(rank + 1 + i) & 7];
  a.rank = rank;
  a.pad = 0;
  hipLaunchKernelGGL(a2a_gemm_relu, dim3(NWG), dim3(NTHREADS), 0, stream, a);
  HIP_CHECK(hipGetLastError());
}
```

```cpp
#include <hip/hip_runtime.h>
#include <rccl/rccl.h>
#include <cstdio>
#include <cstdint>

#define HIP_CHECK(expr) do { hipError_t _e = (expr); if (_e != hipSuccess) { \
    fprintf(stderr, "HIP error %s at %s:%d\n", hipGetErrorString(_e), __FILE__, __LINE__); \
    return; } } while (0)

typedef _Float16 h2 __attribute__((ext_vector_type(2)));
typedef _Float16 h8 __attribute__((ext_vector_type(8)));
typedef float f2 __attribute__((ext_vector_type(2)));
typedef float f4 __attribute__((ext_vector_type(4)));
typedef unsigned u2 __attribute__((ext_vector_type(2)));
typedef unsigned u4 __attribute__((ext_vector_type(4)));
typedef __attribute__((address_space(1))) unsigned gu32;
typedef __attribute__((address_space(4))) const unsigned cu32;

constexpr int NR = 8;
constexpr int KD = 2048, ND = 2048;
constexpr int MS = 256;
constexpr int KS = 256;
constexpr int NWG = 256;
constexpr int NTHREADS = 576;

#define AMIN 3
#define ASPAN 6
#define SPREAD 1
#define WARM 0
#define MTMAJOR 0
#define OUT_AUX 16

constexpr unsigned CNT_OFF = 0;
constexpr unsigned TMO_OFF = 65536;
constexpr unsigned PROF_OFF = 131072;
constexpr unsigned XG_OFF = 1u << 20;
constexpr unsigned XG_BYTES = 1u << 20;
constexpr unsigned WS_NEEDED = XG_OFF + 3 * XG_BYTES;
constexpr unsigned long long DEADLINE_TICKS = 50000000ull;

struct Args {
  const float* x;
  const float* w;
  float* out;
  char* myws;
  char* wsrot[NR];
  int rank;
  int pad;
};

#define STAMP(i) do {} while (0)

__device__ __forceinline__ unsigned cvt2(float a, float b) {
  f2 v = {a, b};
  return __builtin_bit_cast(unsigned, __builtin_convertvector(v, h2));
}
__device__ __forceinline__ unsigned dpp_hi(unsigned old, unsigned src) { return __builtin_amdgcn_update_dpp(old, src, 0x128, 0xf, 0xc, false); }
__device__ __forceinline__ unsigned dpp_lo(unsigned old, unsigned src) { return __builtin_amdgcn_update_dpp(old, src, 0x128, 0xf, 0x3, false); }
__device__ __forceinline__ unsigned min3u(unsigned a, unsigned b, unsigned c) { return min(min(a, b), c); }

__global__ __launch_bounds__(NTHREADS, 3) void a2a_gemm_relu(Args a) {
  __shared__ __attribute__((aligned(16))) char red[8 * 8192];
  STAMP(0);
  const int tid = threadIdx.x;
  const int lane = tid & 63;
  const int wave = __builtin_amdgcn_readfirstlane(tid >> 6);
  const unsigned b = blockIdx.x;
  const int c16 = lane & 15, q = lane >> 4;

  const unsigned xl = b & 7, j = b >> 3;
  const unsigned apos = AMIN + (j * ASPAN) / 32;
  if (__builtin_expect(wave == 0, 1)) {
    __builtin_amdgcn_s_setprio(3);
    const unsigned rank = (unsigned)a.rank;
    const unsigned cidx = (xl - (xl > 3 ? 1 : 0)) * 32 + j;
    const unsigned d = xl == 3 ? 7u : cidx % 7, u = xl == 3 ? j : cidx / 7;
    const unsigned m = u & 3, s = u >> 2;
    const unsigned r = (rank + 1 + d) & 7;
    __amdgpu_buffer_rsrc_t sr = __builtin_amdgcn_make_buffer_rsrc((void*)a.x, 0, KD * KS * 4, 0x00020000);
    const int svoff = (c16 * KS + 8 * q) * 4, sbase = (int)(((MS * r + 64 * m) * KS + 32 * s) * 4);
    f4 v[4][2];
#pragma unroll
    for (int mt = 0; mt < 4; ++mt) {
      v[mt][0] = __builtin_bit_cast(f4, __builtin_amdgcn_raw_buffer_load_b128(sr, svoff, sbase + mt * 16 * KS * 4, 0));
      v[mt][1] = __builtin_bit_cast(f4, __builtin_amdgcn_raw_buffer_load_b128(sr, svoff + 16, sbase + mt * 16 * KS * 4, 0));
    }
    __builtin_amdgcn_sched_barrier(0);
    __builtin_amdgcn_s_barrier();
    STAMP(1);
    const unsigned call = *(cu32*)(a.myws + CNT_OFF + b * 128) + 1;
    char* dws = a.wsrot[d];
    asm volatile("" : "+s"(dws));
    __builtin_amdgcn_sched_barrier(0);
    const unsigned cur = call % 3, old = (call + 2) % 3;
    const unsigned doff = XG_OFF + cur * XG_BYTES + ((rank * 4 + m) * 8 + s) * 4096;
    unsigned t0 = 0, t1 = 0, t2 = 0;
    char* dst = dws + doff + lane * 16;
    char* zer = a.myws + XG_OFF + old * XG_BYTES + b * 4096 + lane * 16;
    if (apos > 8) __builtin_amdgcn_s_barrier();
    u4 h[4];
#pragma unroll
    for (int mt = 0; mt < 4; ++mt) {
      h[mt].x = cvt2(v[mt][0].x, v[mt][0].y);
      h[mt].y = cvt2(v[mt][0].z, v[mt][0].w);
      h[mt].z = cvt2(v[mt][1].x, v[mt][1].y);
      h[mt].w = cvt2(v[mt][1].z, v[mt][1].w);
#pragma unroll
      for (int i = 0; i < 4; ++i) h[mt][i] = h[mt][i] == 0u ? 0x80000000u : h[mt][i];
      asm volatile("" : "+v"(h[mt]));
    }
    const u4 z = {0u, 0u, 0u, 0u};
    STAMP(2);
    __builtin_amdgcn_s_barrier();
    asm volatile("global_store_dwordx4 %0, %1, off sc0 sc1\n\t"
                 "global_store_dwordx4 %0, %2, off offset:1024 sc0 sc1\n\t"
                 "global_store_dwordx4 %0, %3, off offset:2048 sc0 sc1\n\t"
                 "global_store_dwordx4 %0, %4, off offset:3072 sc0 sc1\n\t"
                 "global_store_dwordx4 %5, %6, off sc1\n\t"
                 "global_store_dwordx4 %5, %6, off offset:1024 sc1\n\t"
                 "global_store_dwordx4 %5, %6, off offset:2048 sc1\n\t"
                 "global_store_dwordx4 %5, %6, off offset:3072 sc1\n\t"
                 "s_nop 1"
                 :: "v"(dst), "v"(h[0]), "v"(h[1]), "v"(h[2]), "v"(h[3]), "v"(zer), "v"(z) : "memory");
    __builtin_amdgcn_s_barrier();
    return;
  }

  const unsigned p = (unsigned)wave - 1;
  const unsigned m = j & 3, ns = j >> 2;
  const unsigned n0 = 32 * xl + 256 * ns;
  const unsigned rot = 2 * m + ns;

  __amdgpu_buffer_rsrc_t wr = __builtin_amdgcn_make_buffer_rsrc((void*)a.w, 0, KD * ND * 4, 0x00020000);
  const int wvoff = ((8 * q + 2 * ((lane >> 3) & 1)) * ND + 4 * (lane & 7)) * 4;
  const int wbase = (KS * p * ND + n0) * 4;
  auto wload = [&](int n) -> f4 {
    const int soff = wbase + (int)((((n >> 2) + rot) & 7) * 32 + ((n & 1) + 4 * ((n >> 1) & 1))) * ND * 4;
    return __builtin_bit_cast(f4, __builtin_amdgcn_raw_buffer_load_b128(wr, wvoff, soff, 0));
  };
  f4 raw[32];
  const unsigned call = *(cu32*)(a.myws + CNT_OFF + b * 128) + 1;
  __builtin_amdgcn_s_barrier();
  STAMP(1);
#pragma unroll
  for (int n = 0; n < 12; ++n) {
    if (n >= AMIN && n < AMIN + ASPAN && n == (int)apos) { __builtin_amdgcn_s_barrier(); __builtin_amdgcn_s_barrier(); }
    if (n == 8) __builtin_amdgcn_s_barrier();
    raw[n] = wload(n);
    __builtin_amdgcn_sched_barrier(0);
  }
  char* xg = a.myws + XG_OFF + (call % 3) * XG_BYTES + (p * 4 + m) * 32768;
  __amdgpu_buffer_rsrc_t xr = __builtin_amdgcn_make_buffer_rsrc((void*)xg, 0, 32768, 0x00020000);
  const int xvoff = lane * 16;
  auto fslot = [](int f) { return MTMAJOR ? (f < 4 ? 0 : 1 + (f - 4) % 7) : f >> 2; };
  auto fmt = [](int f) { return MTMAJOR ? (f < 4 ? f : (f - 4) / 7) : f & 3; };
  auto xload = [&](int i, int k) -> u4 {
    return __builtin_amdgcn_raw_buffer_load_b128(xr, xvoff + fmt(4 * i + k) * 1024, (int)(((fslot(4 * i + k) + rot) & 7) * 4096), 17);
  };
  h8 wf[8][2];
  u4 xf[8][4];
#pragma unroll
  for (int i = 0; i < 8; ++i) {
    u4 f0, f1;
#pragma unroll
    for (int h = 0; h < 2; ++h) {
      const f4 r0 = raw[4 * i + 2 * h], r1 = raw[4 * i + 2 * h + 1];
      const unsigned p0 = cvt2(r0.x, r1.x), p1 = cvt2(r0.y, r1.y), p2 = cvt2(r0.z, r1.z), p3 = cvt2(r0.w, r1.w);
      f0[2 * h] = dpp_hi(p0, p2); f0[2 * h + 1] = dpp_lo(p2, p0);
      f1[2 * h] = dpp_hi(p1, p3); f1[2 * h + 1] = dpp_lo(p3, p1);
    }
    wf[i][0] = __builtin_bit_cast(h8, f0);
    wf[i][1] = __builtin_bit_cast(h8, f1);
    asm volatile("" : "+v"(wf[i][0]), "+v"(wf[i][1]));
    if (i == 1) STAMP(2);
    if (i < 5) {
#pragma unroll
      for (int k = 0; k < 4; ++k) raw[12 + 4 * i + k] = wload(12 + 4 * i + k);
    } else {
#pragma unroll
      for (int mt = 0; mt < 4; ++mt) xf[i - 5][mt] = xload(i - 5, mt);
    }
    __builtin_amdgcn_sched_barrier(0);
  }
  STAMP(3);

  bool timed_out = false;
  f4 acc[4][2];
#pragma unroll
  for (int mt = 0; mt < 4; ++mt) { acc[mt][0] = (f4)0.f; acc[mt][1] = (f4)0.f; }
  char* redw = red + p * 8192 + c16 * 128 + 8 * ((q >> 1) ^ (c16 >> 3));
  auto put = [&](int mt) {
#pragma unroll
    for (int i = 0; i < 4; ++i) {
      f2 pr = {acc[mt][0][i], acc[mt][1][i]};
      if (timed_out) pr = (f2)__builtin_nanf("");
      *(f2*)(redw + mt * 2048 + (((4 * (q & 1) + i) ^ (c16 & 7)) * 16)) = pr;
    }
  };
#pragma unroll
  for (int i = 0; i < 8; ++i) {
    auto missing = [&]() {
      unsigned mn = min3u(xf[i][0].x, xf[i][0].y, xf[i][0].z);
      mn = min3u(mn, xf[i][0].w, xf[i][1].x); mn = min3u(mn, xf[i][1].y, xf[i][1].z);
      mn = min3u(mn, xf[i][1].w, xf[i][2].x); mn = min3u(mn, xf[i][2].y, xf[i][2].z);
      mn = min3u(mn, xf[i][2].w, xf[i][3].x); mn = min3u(mn, xf[i][3].y, xf[i][3].z);
      mn = min(mn, xf[i][3].w);
      return __builtin_amdgcn_ballot_w64(mn == 0u) != 0ull;
    };
    if (__builtin_expect(missing() && !timed_out, 0)) {
      gu32* tmo = (gu32*)(a.myws + TMO_OFF);
      const unsigned long long t0 = __builtin_amdgcn_s_memrealtime();
      for (;;) {
        __builtin_amdgcn_s_sleep(2);
        asm volatile("" ::: "memory");
#pragma unroll
        for (int mt = 0; mt < 4; ++mt) xf[i][mt] = xload(i, mt);
        if (!missing()) break;
        if (__hip_atomic_load(tmo, __ATOMIC_RELAXED, __HIP_MEMORY_SCOPE_AGENT) == call ||
            __builtin_amdgcn_s_memrealtime() - t0 > DEADLINE_TICKS) {
          __hip_atomic_store(tmo, call, __ATOMIC_RELAXED, __HIP_MEMORY_SCOPE_AGENT);
          timed_out = true;
          break;
        }
      }
    }
    if (i == 0) STAMP(4);
#pragma unroll
    for (int k = 0; k < 4; ++k) {
      const int sl = fslot(4 * i + k), mt = fmt(4 * i + k);
      acc[mt][0] = __builtin_amdgcn_mfma_f32_16x16x32_f16(wf[sl][0], __builtin_bit_cast(h8, xf[i][k]), acc[mt][0], 0, 0, 0);
      acc[mt][1] = __builtin_amdgcn_mfma_f32_16x16x32_f16(wf[sl][1], __builtin_bit_cast(h8, xf[i][k]), acc[mt][1], 0, 0, 0);
    }
    if (i + 3 < 8) {
#pragma unroll
      for (int k = 0; k < 4; ++k) xf[i + 3][k] = xload(i + 3, k);
    }
    __builtin_amdgcn_sched_barrier(0);
    if (MTMAJOR) {
      if (i == 2) put(0);
      if (i == 4) put(1);
      if (i == 6) put(2);
      __builtin_amdgcn_sched_barrier(0);
    }
  }
  if (!MTMAJOR) { put(0); put(1); put(2); }
  put(3);
  __syncthreads();
  {
    const int row = 8 * p + (lane >> 3), seg = lane & 7;
    const char* rp = red + row * 128 + ((seg ^ (row & 7)) * 16);
    f4 sum = *(const f4*)rp;
#pragma unroll
    for (int w = 1; w < 8; ++w) sum += *(const f4*)(rp + w * 8192);
    if (p & 1) sum = sum.zwxy;
    f4 o;
#pragma unroll
    for (int i = 0; i < 4; ++i) o[i] = __builtin_elementwise_maximum(sum[i], 0.0f);
    __amdgpu_buffer_rsrc_t orr = __builtin_amdgcn_make_buffer_rsrc((void*)a.out, 0, MS * ND * 4, 0x00020000);
    __builtin_amdgcn_raw_buffer_store_b128(__builtin_bit_cast(u4, o), orr, (row * ND + 4 * seg) * 4, (int)((64 * m * ND + n0) * 4), OUT_AUX);
  }
  if (tid == 64) __hip_atomic_store((gu32*)(a.myws + CNT_OFF + b * 128), call, __ATOMIC_RELAXED, __HIP_MEMORY_SCOPE_AGENT);
}

extern "C" void kernel_launch(int rank, int nranks, ncclComm_t comm, hipStream_t stream,
                              void* const* d_in, const int* in_sizes, int n_in,
                              void* d_out, int out_size,
                              const int* in_dtypes, int out_dtype,
                              void* const* d_out_all,
                              void* const* d_ws_all, size_t ws_bytes) {
  if (nranks != NR || (unsigned)rank >= (unsigned)NR || n_in < 2 || ws_bytes < WS_NEEDED ||
      in_sizes[0] != KD * KS || in_sizes[1] != KD * ND || out_size != MS * ND) {
    fprintf(stderr, "kernel_launch: unexpected shapes\n");
    return;
  }
  Args a;
  a.x = (const float*)d_in[0];
  a.w = (const float*)d_in[1];
  a.out = (float*)d_out;
  a.myws = (char*)d_ws_all[rank];
  for (int i = 0; i < NR; ++i) a.wsrot[i] = (char*)d_ws_all[(rank + 1 + i) & 7];
  a.rank = rank;
  a.pad = 0;
  hipLaunchKernelGGL(a2a_gemm_relu, dim3(NWG), dim3(NTHREADS), 0, stream, a);
  HIP_CHECK(hipGetLastError());
}
```

```cpp
#include <hip/hip_runtime.h>
#include <rccl/rccl.h>
#include <cstddef>
#include <cstdio>

#define HIP_CHECK(expr) do { hipError_t _e = (expr); if (_e != hipSuccess) { \
    fprintf(stderr, "HIP error %s at %s:%d\n", hipGetErrorString(_e), __FILE__, __LINE__); \
    return; } } while (0)

typedef _Float16 h2 __attribute__((ext_vector_type(2)));
typedef _Float16 h8 __attribute__((ext_vector_type(8)));
typedef float f2 __attribute__((ext_vector_type(2)));
typedef float f4 __attribute__((ext_vector_type(4)));
typedef unsigned u4 __attribute__((ext_vector_type(4)));
typedef unsigned u8v __attribute__((ext_vector_type(8)));
typedef __attribute__((address_space(1))) unsigned gu32;
typedef __attribute__((address_space(4))) const unsigned cu32;
typedef __attribute__((address_space(3))) char lchar;

constexpr int NR = 8;
constexpr int KD = 2048, ND = 2048;
constexpr int MS = 256;
constexpr int KS = 256;
constexpr int NWG = 256;
constexpr int NTHREADS = 576;
#define P_AMIN 3
#define P_ASPAN 6
#define POLLI 4
#define DYNP 0
constexpr int AMIN = P_AMIN, ASPAN = P_ASPAN;
constexpr unsigned RESPERM = 0x64201573u;
constexpr int AUX_SYS = 17, AUX_WT = 16;

constexpr unsigned CNT_OFF = 0;
constexpr unsigned FLAG_OFF = 32768;
constexpr unsigned TMO_OFF = 40960;
constexpr unsigned PROF_OFF = 131072;
constexpr unsigned XG_OFF = 1u << 20;
constexpr unsigned XG_BYTES = 1u << 20;
constexpr unsigned WS_NEEDED = XG_OFF + 2 * XG_BYTES;
constexpr unsigned long long DEADLINE_TICKS = 50000000ull;

struct Args {
  const float* x;
  const float* w;
  float* out;
  char* myws;
  int rank;
  int pad;
  char* wsrot[NR];
};
static_assert(offsetof(Args, x) == 0 && offsetof(Args, w) == 8 && offsetof(Args, out) == 16 && offsetof(Args, myws) == 24 &&
              offsetof(Args, rank) == 32, "the kernel reads these by offset");

#define STAMP(i) do {} while (0)
#define SCHED() __builtin_amdgcn_sched_barrier(0)
#define BAR() __builtin_amdgcn_s_barrier()

__device__ __forceinline__ unsigned cvt2(float a, float b) {
  f2 v = {a, b};
  return __builtin_bit_cast(unsigned, __builtin_convertvector(v, h2));
}
__device__ __forceinline__ unsigned dpp_hi(unsigned old, unsigned src) { return __builtin_amdgcn_update_dpp(old, src, 0x128, 0xf, 0xc, false); }
__device__ __forceinline__ unsigned dpp_lo(unsigned old, unsigned src) { return __builtin_amdgcn_update_dpp(old, src, 0x128, 0xf, 0x3, false); }

__global__ __launch_bounds__(NTHREADS, 3) void a2a_gemm_relu(Args a) {
  __shared__ __attribute__((aligned(16))) char red[8 * 8192];
  STAMP(0);
  u8v ka;
  unsigned rank;
  {
    const auto kp = __builtin_amdgcn_kernarg_segment_ptr();
    asm volatile("s_load_dwordx8 %0, %2, 0x0\n\ts_load_dword %1, %2, 0x20" : "=&s"(ka), "=&s"(rank) : "s"(kp));
    SCHED();
  }
  const int tid = threadIdx.x;
  const int lane = tid & 63;
  const int wave = __builtin_amdgcn_readfirstlane(tid >> 6);
  const unsigned b = blockIdx.x;
  const int c16 = lane & 15, q = lane >> 4;
  const unsigned xl = b & 7, j = b >> 3;
  const unsigned m = j & 3, ns = j >> 2;
  const unsigned n0 = 32 * ((RESPERM >> (4 * xl)) & 7) + 256 * ns;
  const bool comm = wave == 0;
  const unsigned p = (unsigned)(wave - 1) & 7;

  const unsigned cidx = (xl - 1) * 32 + j;
  unsigned dq = cidx / 7, dr7 = cidx - 7 * dq;
  asm volatile("" : "+s"(dq), "+s"(dr7));
  const unsigned d = xl == 0 ? 7u : dr7;
  const unsigned u = xl == 0 ? j : dq;
  const unsigned um = u & 3, us = u >> 2;

  const unsigned rot = 2 * m + ns;
  auto kstep = [&](int i) -> unsigned { return (i + rot) & 7; };
  const int wvoff = ((8 * q + 2 * ((lane >> 3) & 1)) * ND + 4 * (lane & 7)) * 4;
  const int wbase = (KS * p * ND + n0) * 4;

  int xb0 = (int)((64 * um * KS + 32 * us) * 4), wb0 = wbase + (int)kstep(0) * 32 * ND * 4, wb1 = wbase + (int)kstep(1) * 32 * ND * 4;
  unsigned ap = AMIN + (j * ASPAN) / 32;
  asm volatile("" : "+s"(xb0), "+s"(wb0), "+s"(wb1), "+s"(ap));
  int fvoff = comm ? (c16 * KS + 8 * q) * 4 : wvoff;
  int fb0 = comm ? xb0 : wb0, fb1 = comm ? xb0 + 32 * KS * 4 : wb1;
  int fsa = comm ? 16 : ND * 4, fsb = comm ? 16 * KS * 4 : 4 * ND * 4;
  unsigned frk = comm ? MS * KS * 4 : 0, fbytes = comm ? KD * KS * 4 : KD * ND * 4, dp1 = d + 1;
  unsigned bpos = comm ? 99u : ap;
  asm volatile("" : "+v"(fvoff));
  asm volatile("" : "+s"(fb0), "+s"(fb1), "+s"(fsa), "+s"(fsb), "+s"(frk), "+s"(fbytes), "+s"(bpos), "+s"(dp1));
  asm volatile("s_waitcnt lgkmcnt(0)" : "+s"(ka), "+s"(rank));
  const float* xp = (const float*)(((unsigned long long)ka[1] << 32) | ka[0]);
  const float* wp = (const float*)(((unsigned long long)ka[3] << 32) | ka[2]);
  float* outp = (float*)(((unsigned long long)ka[5] << 32) | ka[4]);
  char* myws = (char*)(((unsigned long long)ka[7] << 32) | ka[6]);
  f4 raw[32];
  unsigned cntv;
  {
    const int fr0 = (int)(((rank + dp1) & 7) * frk);
    __amdgpu_buffer_rsrc_t fr = __builtin_amdgcn_make_buffer_rsrc((void*)(comm ? xp : wp), 0, fbytes, 0x00020000);
    SCHED();
    if (!comm) BAR();
    STAMP(1);
#pragma unroll
    for (int n = 0; n < 8; ++n) {
      if (n >= AMIN && n == (int)bpos) { BAR(); BAR(); }
      const int soff = fr0 + (n < 4 ? fb0 : fb1) + (n & 1) * fsa + ((n >> 1) & 1) * fsb;
      raw[n] = __builtin_bit_cast(f4, __builtin_amdgcn_raw_buffer_load_b128(fr, fvoff, soff, 0));
      if (n == 1) cntv = *(cu32*)(myws + CNT_OFF + b * 128);
      SCHED();
    }
  }

  if (__builtin_expect(comm, 1)) {
    BAR();
    SCHED();
    __builtin_amdgcn_s_setprio(3);
    char* dws = a.wsrot[d];
    unsigned cv = cntv;
    asm volatile("" : "+s"(dws), "+s"(cv));
    SCHED();
    const unsigned call = cv + 1;
    __amdgpu_buffer_rsrc_t dr = __builtin_amdgcn_make_buffer_rsrc((void*)(dws + XG_OFF + (call & 1) * XG_BYTES), 0, XG_BYTES, 0x00020000);
    const int dbase = (int)(((rank * 4 + um) * 8 + us) * 4096);
    gu32* flag = (gu32*)(dws + FLAG_OFF + (rank * 4 + um) * 128 + us * 4);
    u4 h[4];
#pragma unroll
    for (int t = 0; t < 4; ++t) {
      const f4 lo = raw[2 * t], hi = raw[2 * t + 1];
      h[t] = u4{cvt2(lo.x, lo.y), cvt2(lo.z, lo.w), cvt2(hi.x, hi.y), cvt2(hi.z, hi.w)};
      asm volatile("" : "+v"(h[t]));
    }
    STAMP(2);
    SCHED();
    BAR();
#pragma unroll
    for (int t = 0; t < 4; ++t) __builtin_amdgcn_raw_buffer_store_b128(h[t], dr, lane * 16 + t * 1024, dbase, AUX_SYS);
    SCHED();
    BAR();
    BAR();
    asm volatile("s_waitcnt vmcnt(0)" ::: "memory");
    if (lane == 0) __hip_atomic_store(flag, call, __ATOMIC_RELAXED, __HIP_MEMORY_SCOPE_SYSTEM);
    asm volatile("s_endpgm" ::: "memory");
  }

  if (DYNP) __builtin_amdgcn_s_setprio(2);
  __amdgpu_buffer_rsrc_t wr = __builtin_amdgcn_make_buffer_rsrc((void*)wp, 0, KD * ND * 4, 0x00020000);
  auto wload = [&](unsigned s, int k) -> f4 {
    const int soff = wbase + (int)(s * 32 + (k & 1) + 4 * (k >> 1)) * ND * 4;
    return __builtin_bit_cast(f4, __builtin_amdgcn_raw_buffer_load_b128(wr, wvoff, soff, 0));
  };
  if (bpos == 8) { BAR(); BAR(); }
  BAR();
#pragma unroll
  for (int n = 8; n < 12; ++n) raw[n] = wload(kstep(2), n & 3);
  SCHED();
  asm volatile("" : "+s"(cntv));
  const unsigned call = cntv + 1;
  __amdgpu_buffer_rsrc_t xr = __builtin_amdgcn_make_buffer_rsrc((void*)(myws + XG_OFF + (call & 1) * XG_BYTES + (p * 4 + m) * 32768), 0, 32768, 0x00020000);
  const int xvoff = lane * 16;

  gu32* fl = (gu32*)(myws + FLAG_OFF + (p * 4 + m) * 128) + (lane & 7);
  unsigned have = 0, fv = 0;
  bool timed_out = false;
  auto arrived = [&](unsigned f) -> unsigned { return (unsigned)__builtin_amdgcn_ballot_w64((int)(f - call) >= 0) & 255u; };
  u4 xf[8][4];
  auto xunit = [&](int i) {
    const unsigned s = kstep(i);
    if (__builtin_expect(!((have >> s) & 1), 0)) {
      gu32* tmo = (gu32*)(myws + TMO_OFF);
      const unsigned long long t0 = __builtin_amdgcn_s_memrealtime();
      for (;;) {
        have = arrived(__hip_atomic_load(fl, __ATOMIC_RELAXED, __HIP_MEMORY_SCOPE_SYSTEM));
        if ((have >> s) & 1) break;
        if (__hip_atomic_load(tmo, __ATOMIC_RELAXED, __HIP_MEMORY_SCOPE_AGENT) == call ||
            __builtin_amdgcn_s_memrealtime() - t0 > DEADLINE_TICKS) {
          __hip_atomic_store(tmo, call, __ATOMIC_RELAXED, __HIP_MEMORY_SCOPE_AGENT);
          timed_out = true;
          have = 255;
          break;
        }
        __builtin_amdgcn_s_sleep(2);
      }
    }
    asm volatile("" ::: "memory");
    SCHED();
#pragma unroll
    for (int mt = 0; mt < 4; ++mt) {
      xf[i][mt] = __builtin_amdgcn_raw_buffer_load_b128(xr, xvoff + mt * 1024, (int)(s * 4096), AUX_SYS);
      SCHED();
    }
  };

  h8 wf[8][2];
#pragma unroll
  for (int i = 0; i < 8; ++i) {
    u4 f0, f1;
#pragma unroll
    for (int h = 0; h < 2; ++h) {
      const f4 r0 = raw[4 * i + 2 * h], r1 = raw[4 * i + 2 * h + 1];
      const unsigned p0 = cvt2(r0.x, r1.x), p1 = cvt2(r0.y, r1.y), p2 = cvt2(r0.z, r1.z), p3 = cvt2(r0.w, r1.w);
      f0[2 * h] = dpp_hi(p0, p2); f0[2 * h + 1] = dpp_lo(p2, p0);
      f1[2 * h] = dpp_hi(p1, p3); f1[2 * h + 1] = dpp_lo(p3, p1);
    }
    wf[i][0] = __builtin_bit_cast(h8, f0);
    wf[i][1] = __builtin_bit_cast(h8, f1);
    asm volatile("" : "+v"(wf[i][0]), "+v"(wf[i][1]));
    if (i == 1) STAMP(2);
    if (i == POLLI) { fv = __hip_atomic_load(fl, __ATOMIC_RELAXED, __HIP_MEMORY_SCOPE_SYSTEM); SCHED(); }
    if (i == 5) have = arrived(fv);
    if (i < 5) {
#pragma unroll
      for (int k = 0; k < 4; ++k) raw[12 + 4 * i + k] = wload(kstep(3 + i), k);
    } else {
      xunit(i - 5);
    }
    SCHED();
  }
  STAMP(3);

  f4 acc[4][2];
#pragma unroll
  for (int mt = 0; mt < 4; ++mt) { acc[mt][0] = (f4)0.f; acc[mt][1] = (f4)0.f; }
  int wa[4], ra, ovoff, osoff;
#pragma unroll
  for (int i = 0; i < 8; ++i) {
    if (DYNP && i == DYNP - 1) __builtin_amdgcn_s_setprio(0);
    if (i + 3 < 8) xunit(i + 3);
#pragma unroll
    for (int mt = 0; mt < 4; ++mt) {
      acc[mt][0] = __builtin_amdgcn_mfma_f32_16x16x32_f16(wf[i][0], __builtin_bit_cast(h8, xf[i][mt]), acc[mt][0], 0, 0, 0);
      acc[mt][1] = __builtin_amdgcn_mfma_f32_16x16x32_f16(wf[i][1], __builtin_bit_cast(h8, xf[i][mt]), acc[mt][1], 0, 0, 0);
    }
    if (i == 5) {
      const int wb = (int)p * 8192 + c16 * 128 + 8 * ((q >> 1) ^ (c16 >> 3));
#pragma unroll
      for (int e = 0; e < 4; ++e) { wa[e] = wb + (((4 * (q & 1) + e) ^ (c16 & 7)) * 16); asm volatile("" : "+v"(wa[e])); }
      const int row = 8 * (int)p + (lane >> 3), seg = lane & 7;
      ra = row * 128 + ((seg ^ (row & 7)) * 16);
      ovoff = (row * ND + 4 * seg) * 4;
      osoff = (int)((64 * m * ND + n0) * 4);
      asm volatile("" : "+v"(ra), "+v"(ovoff));
      asm volatile("" : "+s"(osoff));
    }
    if (i == 0) STAMP(4);
    SCHED();
  }
  if (__builtin_expect(timed_out, 0)) {
#pragma unroll
    for (int mt = 0; mt < 4; ++mt) { acc[mt][0] = (f4)__builtin_nanf(""); acc[mt][1] = acc[mt][0]; }
  }
#pragma unroll
  for (int mt = 0; mt < 4; mt += 2) {
#pragma unroll
    for (int e = 0; e < 4; ++e) {
      *(__attribute__((address_space(3))) f2*)((lchar*)red + wa[e] + mt * 2048) = f2{acc[mt][0][e], acc[mt][1][e]};
      *(__attribute__((address_space(3))) f2*)((lchar*)red + wa[e] + mt * 2048 + 2048) = f2{acc[mt + 1][0][e], acc[mt + 1][1][e]};
    }
    SCHED();
  }
  __amdgpu_buffer_rsrc_t orr = __builtin_amdgcn_make_buffer_rsrc((void*)outp, 0, MS * ND * 4, 0x00020000);
  __syncthreads();
  {
    f4 v[8];
#pragma unroll
    for (int w = 0; w < 8; ++w) v[w] = *(const __attribute__((address_space(3))) f4*)((lchar*)red + ra + w * 8192);
    SCHED();
    f4 sum = ((v[0] + v[1]) + (v[2] + v[3])) + ((v[4] + v[5]) + (v[6] + v[7]));
    if (p & 1) sum = sum.zwxy;
    f4 o;
#pragma unroll
    for (int e = 0; e < 4; ++e) o[e] = __builtin_elementwise_maximum(sum[e], 0.0f);
    __builtin_amdgcn_raw_buffer_store_b128(__builtin_bit_cast(u4, o), orr, ovoff, osoff, AUX_WT);
  }
  if (tid == 64) __hip_atomic_store((gu32*)(myws + CNT_OFF + b * 128), call, __ATOMIC_RELAXED, __HIP_MEMORY_SCOPE_AGENT);
}

extern "C" void kernel_launch(int rank, int nranks, ncclComm_t comm, hipStream_t stream,
                              void* const* d_in, const int* in_sizes, int n_in,
                              void* d_out, int out_size,
                              const int* in_dtypes, int out_dtype,
                              void* const* d_out_all,
                              void* const* d_ws_all, size_t ws_bytes) {
  if (nranks != NR || (unsigned)rank >= (unsigned)NR || n_in < 2 || ws_bytes < WS_NEEDED ||
      in_sizes[0] != KD * KS || in_sizes[1] != KD * ND || out_size != MS * ND) {
    fprintf(stderr, "kernel_launch: unexpected shapes\n");
    return;
  }
  Args a;
  a.x = (const float*)d_in[0];
  a.w = (const float*)d_in[1];
  a.out = (float*)d_out;
  a.myws = (char*)d_ws_all[rank];
  a.rank = rank;
  a.pad = 0;
  for (int i = 0; i < NR; ++i) a.wsrot[i] = (char*)d_ws_all[(rank + 1 + i) & 7];
  hipLaunchKernelGGL(a2a_gemm_relu, dim3(NWG), dim3(NTHREADS), 0, stream, a);
  HIP_CHECK(hipGetLastError());
}
```

```cpp
#include <hip/hip_runtime.h>
#include <rccl/rccl.h>
#include <cstddef>
#include <cstdio>

#define HIP_CHECK(expr) do { hipError_t _e = (expr); if (_e != hipSuccess) { \
    fprintf(stderr, "HIP error %s at %s:%d\n", hipGetErrorString(_e), __FILE__, __LINE__); \
    return; } } while (0)

typedef _Float16 h2 __attribute__((ext_vector_type(2)));
typedef _Float16 h8 __attribute__((ext_vector_type(8)));
typedef float f2 __attribute__((ext_vector_type(2)));
typedef float f4 __attribute__((ext_vector_type(4)));
typedef unsigned u4 __attribute__((ext_vector_type(4)));
typedef unsigned u8v __attribute__((ext_vector_type(8)));
typedef __attribute__((address_space(1))) unsigned gu32;
typedef __attribute__((address_space(4))) const unsigned cu32;
typedef __attribute__((address_space(3))) char lchar;

constexpr int NR = 8;
constexpr int KD = 2048, ND = 2048;
constexpr int MS = 256;
constexpr int KS = 256;
constexpr int NWG = 256;
constexpr int NTHREADS = 576;
#define P_AMIN 3
#define P_ASPAN 6
#define SLEEP 1
constexpr int AMIN = P_AMIN, ASPAN = P_ASPAN;
constexpr unsigned RESPERM = 0x64201573u;
constexpr int AUX_SYS = 17, AUX_WT = 16;

constexpr unsigned CNT_OFF = 0;
constexpr unsigned FLAG_OFF = 32768;
constexpr unsigned TMO_OFF = 40960;
constexpr unsigned PROF_OFF = 131072;
constexpr unsigned XG_OFF = 1u << 20;
constexpr unsigned XG_BYTES = 1u << 20;
constexpr unsigned WS_NEEDED = XG_OFF + 3 * XG_BYTES;
constexpr unsigned long long DEADLINE_TICKS = 50000000ull;

struct Args {
  const float* x;
  const float* w;
  float* out;
  char* myws;
  int rank;
  int pad;
  char* wsrot[NR];
};
static_assert(offsetof(Args, x) == 0 && offsetof(Args, w) == 8 && offsetof(Args, out) == 16 && offsetof(Args, myws) == 24 &&
              offsetof(Args, rank) == 32, "the kernel reads these by offset");

#define STAMP(i) do {} while (0)
#define SCHED() __builtin_amdgcn_sched_barrier(0)
#define BAR() __builtin_amdgcn_s_barrier()

__device__ __forceinline__ unsigned cvt2(float a, float b) {
  f2 v = {a, b};
  return __builtin_bit_cast(unsigned, __builtin_convertvector(v, h2));
}
__device__ __forceinline__ unsigned dpp_hi(unsigned old, unsigned src) { return __builtin_amdgcn_update_dpp(old, src, 0x128, 0xf, 0xc, false); }
__device__ __forceinline__ unsigned dpp_lo(unsigned old, unsigned src) { return __builtin_amdgcn_update_dpp(old, src, 0x128, 0xf, 0x3, false); }
__device__ __forceinline__ bool any_zero(const u4* f) {
  unsigned mn = min(min(f[0].x, f[0].y), f[0].z);
  mn = min(min(mn, f[0].w), f[1].x); mn = min(min(mn, f[1].y), f[1].z);
  mn = min(min(mn, f[1].w), f[2].x); mn = min(min(mn, f[2].y), f[2].z);
  mn = min(min(mn, f[2].w), f[3].x); mn = min(min(mn, f[3].y), f[3].z);
  mn = min(mn, f[3].w);
  return __builtin_amdgcn_ballot_w64(mn == 0u) != 0ull;
}

__global__ __launch_bounds__(NTHREADS, 3) void a2a_gemm_relu(Args a) {
  __shared__ __attribute__((aligned(16))) char red[8 * 8192];
  STAMP(0);
  u8v ka;
  unsigned rank;
  {
    const auto kp = __builtin_amdgcn_kernarg_segment_ptr();
    asm volatile("s_load_dwordx8 %0, %2, 0x0\n\ts_load_dword %1, %2, 0x20" : "=&s"(ka), "=&s"(rank) : "s"(kp));
    SCHED();
  }
  const int tid = threadIdx.x;
  const int lane = tid & 63;
  const int wave = __builtin_amdgcn_readfirstlane(tid >> 6);
  const unsigned b = blockIdx.x;
  const int c16 = lane & 15, q = lane >> 4;
  const unsigned xl = b & 7, j = b >> 3;
  const unsigned m = j & 3, ns = j >> 2;
  const unsigned n0 = 32 * ((RESPERM >> (4 * xl)) & 7) + 256 * ns;
  const bool comm = wave == 0;
  const unsigned p = (unsigned)(wave - 1) & 7;

  auto kstep = [&](int i) -> unsigned { return (i + 2 * m) & 7; };
  unsigned dr7 = (xl - 1 + j) % 7;
  asm volatile("" : "+s"(dr7));
  const unsigned d = xl == 0 ? 7u : dr7;
  const unsigned us = (ns + 2 * m) & 7;
  const int wvoff = ((8 * q + 2 * ((lane >> 3) & 1)) * ND + 4 * (lane & 7)) * 4;
  const int wbase = (KS * p * ND + n0) * 4;

  int xb0 = (int)((64 * m * KS + 32 * us) * 4), wb0 = wbase + (int)kstep(0) * 32 * ND * 4, wb1 = wbase + (int)kstep(1) * 32 * ND * 4;
  unsigned ap = AMIN + (j * ASPAN) / 32;
  asm volatile("" : "+s"(xb0), "+s"(wb0), "+s"(wb1), "+s"(ap));
  int fvoff = comm ? (c16 * KS + 8 * q) * 4 : wvoff;
  int fb0 = comm ? xb0 : wb0, fb1 = comm ? xb0 + 32 * KS * 4 : wb1;
  int fsa = comm ? 16 : ND * 4, fsb = comm ? 16 * KS * 4 : 4 * ND * 4;
  unsigned frk = comm ? MS * KS * 4 : 0, fbytes = comm ? KD * KS * 4 : KD * ND * 4, dp1 = d + 1;
  unsigned bpos = comm ? 99u : ap;
  asm volatile("" : "+v"(fvoff));
  asm volatile("" : "+s"(fb0), "+s"(fb1), "+s"(fsa), "+s"(fsb), "+s"(frk), "+s"(fbytes), "+s"(bpos), "+s"(dp1));
  asm volatile("s_waitcnt lgkmcnt(0)" : "+s"(ka), "+s"(rank));
  const float* xp = (const float*)(((unsigned long long)ka[1] << 32) | ka[0]);
  const float* wp = (const float*)(((unsigned long long)ka[3] << 32) | ka[2]);
  float* outp = (float*)(((unsigned long long)ka[5] << 32) | ka[4]);
  char* myws = (char*)(((unsigned long long)ka[7] << 32) | ka[6]);
  f4 raw[32];
  unsigned cntv;
  {
    const int fr0 = (int)(((rank + dp1) & 7) * frk);
    __amdgpu_buffer_rsrc_t fr = __builtin_amdgcn_make_buffer_rsrc((void*)(comm ? xp : wp), 0, fbytes, 0x00020000);
    SCHED();
    if (!comm) BAR();
    STAMP(1);
#pragma unroll
    for (int n = 0; n < 8; ++n) {
      if (n >= AMIN && n == (int)bpos) { BAR(); BAR(); }
      const int soff = fr0 + (n < 4 ? fb0 : fb1) + (n & 1) * fsa + ((n >> 1) & 1) * fsb;
      raw[n] = __builtin_bit_cast(f4, __builtin_amdgcn_raw_buffer_load_b128(fr, fvoff, soff, 0));
      if (n == 1) cntv = *(cu32*)(myws + CNT_OFF + b * 128);
      SCHED();
    }
  }

  if (__builtin_expect(comm, 1)) {
    BAR();
    SCHED();
    __builtin_amdgcn_s_setprio(3);
    char* dws = a.wsrot[d];
    unsigned cv = cntv;
    asm volatile("" : "+s"(dws), "+s"(cv));
    SCHED();
    const unsigned call = cv + 1;
    __amdgpu_buffer_rsrc_t dr = __builtin_amdgcn_make_buffer_rsrc((void*)(dws + XG_OFF + (call % 3) * XG_BYTES), 0, XG_BYTES, 0x00020000);
    __amdgpu_buffer_rsrc_t zr = __builtin_amdgcn_make_buffer_rsrc((void*)(myws + XG_OFF + ((call + 2) % 3) * XG_BYTES), 0, XG_BYTES, 0x00020000);
    const int dbase = (int)(((rank * 4 + m) * 8 + us) * 4096);
    u4 h[4];
#pragma unroll
    for (int t = 0; t < 4; ++t) {
      const f4 lo = raw[2 * t], hi = raw[2 * t + 1];
      h[t] = u4{cvt2(lo.x, lo.y), cvt2(lo.z, lo.w), cvt2(hi.x, hi.y), cvt2(hi.z, hi.w)};
      asm volatile("" : "+v"(h[t]));
    }
    STAMP(2);
    SCHED();
    BAR();
#pragma unroll
    for (int t = 0; t < 4; ++t) __builtin_amdgcn_raw_buffer_store_b128(h[t], dr, lane * 16 + t * 1024, dbase, AUX_SYS);
#pragma unroll
    for (int t = 0; t < 4; ++t) __builtin_amdgcn_raw_buffer_store_b128(u4{0u, 0u, 0u, 0u}, zr, lane * 16 + t * 1024, (int)(b * 4096), AUX_WT);
    SCHED();
    BAR();
    BAR();
    if (any_zero(h)) {
      asm volatile("s_waitcnt vmcnt(0)" ::: "memory");
      if (lane == 0)
        __hip_atomic_store((gu32*)(dws + FLAG_OFF + (rank * 4 + m) * 128 + ns * 4), call, __ATOMIC_RELAXED, __HIP_MEMORY_SCOPE_SYSTEM);
    }
    asm volatile("s_endpgm" ::: "memory");
  }

  __amdgpu_buffer_rsrc_t wr = __builtin_amdgcn_make_buffer_rsrc((void*)wp, 0, KD * ND * 4, 0x00020000);
  auto wload = [&](unsigned s, int k) -> f4 {
    const int soff = wbase + (int)(s * 32 + (k & 1) + 4 * (k >> 1)) * ND * 4;
    return __builtin_bit_cast(f4, __builtin_amdgcn_raw_buffer_load_b128(wr, wvoff, soff, 0));
  };
  if (bpos == 8) { BAR(); BAR(); }
  BAR();
#pragma unroll
  for (int n = 8; n < 12; ++n) raw[n] = wload(kstep(2), n & 3);
  SCHED();
  asm volatile("" : "+s"(cntv));
  const unsigned call = cntv + 1;
  __amdgpu_buffer_rsrc_t xr = __builtin_amdgcn_make_buffer_rsrc((void*)(myws + XG_OFF + (call % 3) * XG_BYTES + (p * 4 + m) * 32768), 0, 32768, 0x00020000);
  const int xvoff = lane * 16;

  unsigned seen = 255u, flagged = 0, trusted = 0;
  bool timed_out = false;
  const unsigned long long t0 = __builtin_amdgcn_s_memrealtime();
  auto await = [&](int i) {
    const int pvoff = lane < 8 ? (int)(((lane + 2 * m) & 7) * 4096 + 3 * 1024 + xl * 128 + 124) : 0x7fffffff;
    gu32* fl = (gu32*)(myws + FLAG_OFF + (p * 4 + m) * 128) + (lane & 7);
    gu32* tmo = (gu32*)(myws + TMO_OFF);
    for (;;) {
      asm volatile("" ::: "memory");
      const unsigned v = __builtin_amdgcn_raw_buffer_load_b32(xr, pvoff, 0, AUX_SYS);
      const unsigned f = __hip_atomic_load(fl, __ATOMIC_RELAXED, __HIP_MEMORY_SCOPE_SYSTEM);
      const unsigned t = __hip_atomic_load(tmo, __ATOMIC_RELAXED, __HIP_MEMORY_SCOPE_AGENT);
      flagged = (unsigned)__builtin_amdgcn_ballot_w64((int)(f - call) >= 0) & 255u;
      seen = (unsigned)__builtin_amdgcn_ballot_w64(v != 0u) | flagged;
      if (t == call || __builtin_amdgcn_s_memrealtime() - t0 > DEADLINE_TICKS) {
        __hip_atomic_store(tmo, call, __ATOMIC_RELAXED, __HIP_MEMORY_SCOPE_AGENT);
        timed_out = true;
        seen = flagged = trusted = 255u;
      }
      if ((seen >> i) & 1) break;
      __builtin_amdgcn_s_sleep(SLEEP);
    }
  };
  u4 xf[8][4];
  auto xreq = [&](int i) {
    trusted |= flagged & (1u << i);
    asm volatile("" ::: "memory");
    SCHED();
#pragma unroll
    for (int mt = 0; mt < 4; ++mt) {
      xf[i][mt] = __builtin_amdgcn_raw_buffer_load_b128(xr, xvoff + mt * 1024, (int)(kstep(i) * 4096), AUX_SYS);
      SCHED();
    }
  };

  h8 wf[8][2];
#pragma unroll
  for (int i = 0; i < 8; ++i) {
    u4 f0, f1;
#pragma unroll
    for (int h = 0; h < 2; ++h) {
      const f4 r0 = raw[4 * i + 2 * h], r1 = raw[4 * i + 2 * h + 1];
      const unsigned p0 = cvt2(r0.x, r1.x), p1 = cvt2(r0.y, r1.y), p2 = cvt2(r0.z, r1.z), p3 = cvt2(r0.w, r1.w);
      f0[2 * h] = dpp_hi(p0, p2); f0[2 * h + 1] = dpp_lo(p2, p0);
      f1[2 * h] = dpp_hi(p1, p3); f1[2 * h + 1] = dpp_lo(p3, p1);
    }
    wf[i][0] = __builtin_bit_cast(h8, f0);
    wf[i][1] = __builtin_bit_cast(h8, f1);
    asm volatile("" : "+v"(wf[i][0]), "+v"(wf[i][1]));
    if (i == 1) STAMP(2);
    if (i < 5) {
#pragma unroll
      for (int k = 0; k < 4; ++k) raw[12 + 4 * i + k] = wload(kstep(3 + i), k);
    } else {
      xreq(i - 5);
    }
    SCHED();
  }
  STAMP(3);

  f4 acc[4][2];
#pragma unroll
  for (int mt = 0; mt < 4; ++mt) { acc[mt][0] = (f4)0.f; acc[mt][1] = (f4)0.f; }
  int wa[4], ra, ovoff, osoff;
#pragma unroll
  for (int i = 0; i < 8; ++i) {
    if (__builtin_expect(any_zero(xf[i]) && !((trusted >> i) & 1), 0)) {
      do {
        await(i);
#pragma unroll
        for (int e = i; e < i + 3 && e < 8; ++e) {
          if (e > i && !((seen >> e) & 1)) await(e);
          xreq(e);
        }
      } while (any_zero(xf[i]) && !((trusted >> i) & 1));
    }
    if (i + 3 < 8) {
      if (__builtin_expect(!((seen >> (i + 3)) & 1), 0)) await(i + 3);
      xreq(i + 3);
    }
#pragma unroll
    for (int mt = 0; mt < 4; ++mt) {
      acc[mt][0] = __builtin_amdgcn_mfma_f32_16x16x32_f16(wf[i][0], __builtin_bit_cast(h8, xf[i][mt]), acc[mt][0], 0, 0, 0);
      acc[mt][1] = __builtin_amdgcn_mfma_f32_16x16x32_f16(wf[i][1], __builtin_bit_cast(h8, xf[i][mt]), acc[mt][1], 0, 0, 0);
    }
    if (i == 5) {
      const int wb = (int)p * 8192 + c16 * 128 + 8 * ((q >> 1) ^ (c16 >> 3));
#pragma unroll
      for (int e = 0; e < 4; ++e) { wa[e] = wb + (((4 * (q & 1) + e) ^ (c16 & 7)) * 16); asm volatile("" : "+v"(wa[e])); }
      const int row = 8 * (int)p + (lane >> 3), seg = lane & 7;
      ra = row * 128 + ((seg ^ (row & 7)) * 16);
      ovoff = (row * ND + 4 * seg) * 4;
      osoff = (int)((64 * m * ND + n0) * 4);
      asm volatile("" : "+v"(ra), "+v"(ovoff));
      asm volatile("" : "+s"(osoff));
    }
    if (i == 0) STAMP(4);
    SCHED();
  }
  if (__builtin_expect(timed_out, 0)) {
#pragma unroll
    for (int mt = 0; mt < 4; ++mt) { acc[mt][0] = (f4)__builtin_nanf(""); acc[mt][1] = acc[mt][0]; }
  }
#pragma unroll
  for (int mt = 0; mt < 4; mt += 2) {
#pragma unroll
    for (int e = 0; e < 4; ++e) {
      *(__attribute__((address_space(3))) f2*)((lchar*)red + wa[e] + mt * 2048) = f2{acc[mt][0][e], acc[mt][1][e]};
      *(__attribute__((address_space(3))) f2*)((lchar*)red + wa[e] + mt * 2048 + 2048) = f2{acc[mt + 1][0][e], acc[mt + 1][1][e]};
    }
    SCHED();
  }
  __amdgpu_buffer_rsrc_t orr = __builtin_amdgcn_make_buffer_rsrc((void*)outp, 0, MS * ND * 4, 0x00020000);
  __syncthreads();
  {
    f4 v[8];
#pragma unroll
    for (int w = 0; w < 8; ++w) v[w] = *(const __attribute__((address_space(3))) f4*)((lchar*)red + ra + w * 8192);
    SCHED();
    f4 sum = ((v[0] + v[1]) + (v[2] + v[3])) + ((v[4] + v[5]) + (v[6] + v[7]));
    if (p & 1) sum = sum.zwxy;
    f4 o;
#pragma unroll
    for (int e = 0; e < 4; ++e) o[e] = __builtin_elementwise_maximum(sum[e], 0.0f);
    __builtin_amdgcn_raw_buffer_store_b128(__builtin_bit_cast(u4, o), orr, ovoff, osoff, AUX_WT);
  }
  if (tid == 64) __hip_atomic_store((gu32*)(myws + CNT_OFF + b * 128), call, __ATOMIC_RELAXED, __HIP_MEMORY_SCOPE_AGENT);
}

extern "C" void kernel_launch(int rank, int nranks, ncclComm_t comm, hipStream_t stream,
                              void* const* d_in, const int* in_sizes, int n_in,
                              void* d_out, int out_size,
                              const int* in_dtypes, int out_dtype,
                              void* const* d_out_all,
                              void* const* d_ws_all, size_t ws_bytes) {
  if (nranks != NR || (unsigned)rank >= (unsigned)NR || n_in < 2 || ws_bytes < WS_NEEDED ||
      in_sizes[0] != KD * KS || in_sizes[1] != KD * ND || out_size != MS * ND) {
    fprintf(stderr, "kernel_launch: unexpected shapes\n");
    return;
  }
  Args a;
  a.x = (const float*)d_in[0];
  a.w = (const float*)d_in[1];
  a.out = (float*)d_out;
  a.myws = (char*)d_ws_all[rank];
  a.rank = rank;
  a.pad = 0;
  for (int i = 0; i < NR; ++i) a.wsrot[i] = (char*)d_ws_all[(rank + 1 + i) & 7];
  hipLaunchKernelGGL(a2a_gemm_relu, dim3(NWG), dim3(NTHREADS), 0, stream, a);
  HIP_CHECK(hipGetLastError());
}
```

```cpp
#include <hip/hip_runtime.h>
#include <rccl/rccl.h>
#include <cstddef>
#include <cstdio>

#define HIP_CHECK(expr) do { hipError_t _e = (expr); if (_e != hipSuccess) { \
    fprintf(stderr, "HIP error %s at %s:%d\n", hipGetErrorString(_e), __FILE__, __LINE__); \
    return; } } while (0)

typedef _Float16 h2 __attribute__((ext_vector_type(2)));
typedef _Float16 h8 __attribute__((ext_vector_type(8)));
typedef float f2 __attribute__((ext_vector_type(2)));
typedef float f4 __attribute__((ext_vector_type(4)));
typedef unsigned u4 __attribute__((ext_vector_type(4)));
typedef unsigned u8v __attribute__((ext_vector_type(8)));
typedef __attribute__((address_space(1))) unsigned gu32;
typedef __attribute__((address_space(4))) const unsigned cu32;
typedef __attribute__((address_space(3))) char lchar;

constexpr int NR = 8;
constexpr int KD = 2048, ND = 2048;
constexpr int MS = 256;
constexpr int KS = 256;
constexpr int NWG = 256;
constexpr int NTHREADS = 576;
constexpr int AMIN = 3, ASPAN = 6;
constexpr unsigned RESPERM = 0x64201573u;
constexpr int AUX_SYS = 17, AUX_WT = 16;

constexpr unsigned CNT_OFF = 0;
constexpr unsigned FLAG_OFF = 32768;
constexpr unsigned TMO_OFF = 40960;
constexpr unsigned XG_OFF = 1u << 20;
constexpr unsigned XG_BYTES = 1u << 20;
constexpr unsigned WS_NEEDED = XG_OFF + 3 * XG_BYTES;
constexpr unsigned long long DEADLINE_TICKS = 50000000ull;

struct Args {
  const float* x;
  const float* w;
  float* out;
  char* myws;
  int rank;
  int pad;
  char* wsrot[NR];
};
static_assert(offsetof(Args, x) == 0 && offsetof(Args, w) == 8 && offsetof(Args, out) == 16 && offsetof(Args, myws) == 24 &&
              offsetof(Args, rank) == 32, "the kernel reads these by offset");

#define SCHED() __builtin_amdgcn_sched_barrier(0)
#define BAR() __builtin_amdgcn_s_barrier()

__device__ __forceinline__ unsigned cvt2(float a, float b) {
  f2 v = {a, b};
  return __builtin_bit_cast(unsigned, __builtin_convertvector(v, h2));
}
__device__ __forceinline__ unsigned dpp_hi(unsigned old, unsigned src) { return __builtin_amdgcn_update_dpp(old, src, 0x128, 0xf, 0xc, false); }
__device__ __forceinline__ unsigned dpp_lo(unsigned old, unsigned src) { return __builtin_amdgcn_update_dpp(old, src, 0x128, 0xf, 0x3, false); }
__device__ __forceinline__ bool any_zero(const u4* f) {
  unsigned mn = min(min(f[0].x, f[0].y), f[0].z);
  mn = min(min(mn, f[0].w), f[1].x); mn = min(min(mn, f[1].y), f[1].z);
  mn = min(min(mn, f[1].w), f[2].x); mn = min(min(mn, f[2].y), f[2].z);
  mn = min(min(mn, f[2].w), f[3].x); mn = min(min(mn, f[3].y), f[3].z);
  mn = min(mn, f[3].w);
  return __builtin_amdgcn_ballot_w64(mn == 0u) != 0ull;
}

__global__ __launch_bounds__(NTHREADS, 3) void a2a_gemm_relu(Args a) {
  __shared__ __attribute__((aligned(16))) char red[8 * 8192];
  u8v ka;
  unsigned rank;
  {
    const auto kp = __builtin_amdgcn_kernarg_segment_ptr();
    asm volatile("s_load_dwordx8 %0, %2, 0x0\n\ts_load_dword %1, %2, 0x20" : "=&s"(ka), "=&s"(rank) : "s"(kp));
    SCHED();
  }
  const int tid = threadIdx.x;
  const int lane = tid & 63;
  const int wave = __builtin_amdgcn_readfirstlane(tid >> 6);
  const unsigned b = blockIdx.x;
  const int c16 = lane & 15, q = lane >> 4;
  const unsigned xl = b & 7, j = b >> 3;
  const unsigned m = j & 3, ns = j >> 2;
  const unsigned n0 = 32 * ((RESPERM >> (4 * xl)) & 7) + 256 * ns;
  const bool comm = wave == 0;
  const unsigned p = (unsigned)(wave - 1) & 7;

  auto kstep = [&](int i) -> unsigned { return (i + 2 * m) & 7; };
  unsigned dr7 = (xl - 1 + j) % 7;
  asm volatile("" : "+s"(dr7));
  const unsigned d = xl == 0 ? 7u : dr7;
  const unsigned us = (ns + 2 * m) & 7;
  const int wvoff = ((8 * q + 2 * ((lane >> 3) & 1)) * ND + 4 * (lane & 7)) * 4;
  const int wbase = (KS * p * ND + n0) * 4;

  int xb0 = (int)((64 * m * KS + 32 * us) * 4), wb0 = wbase + (int)kstep(0) * 32 * ND * 4, wb1 = wbase + (int)kstep(1) * 32 * ND * 4;
  unsigned ap = AMIN + (j * ASPAN) / 32;
  asm volatile("" : "+s"(xb0), "+s"(wb0), "+s"(wb1), "+s"(ap));
  int fvoff = comm ? (c16 * KS + 8 * q) * 4 : wvoff;
  const int fb0 = comm ? xb0 : wb0, fb1 = comm ? xb0 + 32 * KS * 4 : wb1;
  const int fsa = comm ? 16 : ND * 4, fsb = comm ? 16 * KS * 4 : 4 * ND * 4;
  int fso[8];
#pragma unroll
  for (int n = 0; n < 8; ++n) fso[n] = (n < 4 ? fb0 : fb1) + (n & 1) * fsa + ((n >> 1) & 1) * fsb;
  unsigned frk = comm ? MS * KS * 4 : 0, fbytes = comm ? MS * KS * 4 : KD * ND * 4, dp1 = d + 1, cnto = CNT_OFF + b * 128;
  unsigned bpos = comm ? 99u : ap;
  asm volatile("" : "+v"(fvoff));
  asm volatile("" : "+s"(fso[0]), "+s"(fso[1]), "+s"(fso[2]), "+s"(fso[3]), "+s"(fso[4]), "+s"(fso[5]), "+s"(fso[6]), "+s"(fso[7]));
  asm volatile("" : "+s"(frk), "+s"(fbytes), "+s"(bpos), "+s"(dp1), "+s"(cnto));
  asm volatile("s_waitcnt lgkmcnt(0)" : "+s"(ka), "+s"(rank));
  const float* xp = (const float*)(((unsigned long long)ka[1] << 32) | ka[0]);
  const float* wp = (const float*)(((unsigned long long)ka[3] << 32) | ka[2]);
  float* outp = (float*)(((unsigned long long)ka[5] << 32) | ka[4]);
  char* myws = (char*)(((unsigned long long)ka[7] << 32) | ka[6]);
  f4 raw[32];
  unsigned cntv;
  {
    const char* fbase = (const char*)(comm ? xp : wp) + ((rank + dp1) & 7) * frk;
    asm volatile("" : "+s"(fbase));
    __amdgpu_buffer_rsrc_t fr = __builtin_amdgcn_make_buffer_rsrc((void*)fbase, 0, fbytes, 0x00020000);
    SCHED();
    if (!comm) BAR();
#pragma unroll
    for (int n = 0; n < 8; ++n) {
      if (n >= AMIN && n == (int)bpos) { BAR(); BAR(); }
      raw[n] = __builtin_bit_cast(f4, __builtin_amdgcn_raw_buffer_load_b128(fr, fvoff, fso[n], 0));
      if (n == 2) cntv = *(cu32*)(myws + cnto);
      SCHED();
    }
  }

  if (__builtin_expect(comm, 1)) {
    BAR();
    SCHED();
    __builtin_amdgcn_s_setprio(3);
    char* dws = a.wsrot[d];
    unsigned cv = cntv;
    asm volatile("" : "+s"(dws), "+s"(cv));
    SCHED();
    const unsigned call = cv + 1;
    __amdgpu_buffer_rsrc_t dr = __builtin_amdgcn_make_buffer_rsrc((void*)(dws + XG_OFF + (call % 3) * XG_BYTES), 0, XG_BYTES, 0x00020000);
    __amdgpu_buffer_rsrc_t zr = __builtin_amdgcn_make_buffer_rsrc((void*)(myws + XG_OFF + ((call + 2) % 3) * XG_BYTES), 0, XG_BYTES, 0x00020000);
    const int dbase = (int)(((rank * 4 + m) * 8 + us) * 4096);
    u4 h[4];
#pragma unroll
    for (int t = 0; t < 4; ++t) {
      const f4 lo = raw[2 * t], hi = raw[2 * t + 1];
      h[t] = u4{cvt2(lo.x, lo.y), cvt2(lo.z, lo.w), cvt2(hi.x, hi.y), cvt2(hi.z, hi.w)};
      asm volatile("" : "+v"(h[t]));
    }
    SCHED();
    BAR();
#pragma unroll
    for (int t = 0; t < 4; ++t) __builtin_amdgcn_raw_buffer_store_b128(h[t], dr, lane * 16 + t * 1024, dbase, AUX_SYS);
#pragma unroll
    for (int t = 0; t < 4; ++t) __builtin_amdgcn_raw_buffer_store_b128(u4{0u, 0u, 0u, 0u}, zr, lane * 16 + t * 1024, (int)(b * 4096), AUX_WT);
    SCHED();
    BAR();
    BAR();
    if (any_zero(h)) {
      asm volatile("s_waitcnt vmcnt(0)" ::: "memory");
      if (lane == 0)
        __hip_atomic_store((gu32*)(dws + FLAG_OFF + (rank * 4 + m) * 128 + ns * 4), call, __ATOMIC_RELAXED, __HIP_MEMORY_SCOPE_SYSTEM);
    }
    asm volatile("s_endpgm" ::: "memory");
  }

  __amdgpu_buffer_rsrc_t wr = __builtin_amdgcn_make_buffer_rsrc((void*)wp, 0, KD * ND * 4, 0x00020000);
  auto wload = [&](unsigned s, int k) -> f4 {
    const int soff = wbase + (int)(s * 32 + (k & 1) + 4 * (k >> 1)) * ND * 4;
    return __builtin_bit_cast(f4, __builtin_amdgcn_raw_buffer_load_b128(wr, wvoff, soff, 0));
  };
  if (bpos == 8) { BAR(); BAR(); }
  BAR();
#pragma unroll
  for (int n = 8; n < 12; ++n) raw[n] = wload(kstep(2), n & 3);
  SCHED();
  asm volatile("" : "+s"(cntv));
  const unsigned call = cntv + 1;
  __amdgpu_buffer_rsrc_t xr = __builtin_amdgcn_make_buffer_rsrc((void*)(myws + XG_OFF + (call % 3) * XG_BYTES + (p * 4 + m) * 32768), 0, 32768, 0x00020000);
  const int xvoff = lane * 16;

  unsigned seen = 255u, flagged = 0, trusted = 0;
  bool timed_out = false;
  const unsigned long long t0 = __builtin_amdgcn_s_memrealtime();
  auto await = [&](int i) {
    const int pvoff = lane < 8 ? (int)(((lane + 2 * m) & 7) * 4096 + 3 * 1024 + xl * 128 + 124) : 0x7fffffff;
    gu32* fl = (gu32*)(myws + FLAG_OFF + (p * 4 + m) * 128) + (lane & 7);
    gu32* tmo = (gu32*)(myws + TMO_OFF);
    for (;;) {
      asm volatile("" ::: "memory");
      const unsigned v = __builtin_amdgcn_raw_buffer_load_b32(xr, pvoff, 0, AUX_SYS);
      const unsigned f = __hip_atomic_load(fl, __ATOMIC_RELAXED, __HIP_MEMORY_SCOPE_SYSTEM);
      const unsigned t = __hip_atomic_load(tmo, __ATOMIC_RELAXED, __HIP_MEMORY_SCOPE_AGENT);
      flagged = (unsigned)__builtin_amdgcn_ballot_w64((int)(f - call) >= 0) & 255u;
      seen = (unsigned)__builtin_amdgcn_ballot_w64(v != 0u) | flagged;
      if (t == call || __builtin_amdgcn_s_memrealtime() - t0 > DEADLINE_TICKS) {
        __hip_atomic_store(tmo, call, __ATOMIC_RELAXED, __HIP_MEMORY_SCOPE_AGENT);
        timed_out = true;
        seen = flagged = trusted = 255u;
      }
      if ((seen >> i) & 1) break;
      __builtin_amdgcn_s_sleep(1);
    }
  };
  u4 xf[8][4];
  auto xreq = [&](int i) {
    trusted |= flagged & (1u << i);
    asm volatile("" ::: "memory");
    SCHED();
#pragma unroll
    for (int mt = 0; mt < 4; ++mt) {
      xf[i][mt] = __builtin_amdgcn_raw_buffer_load_b128(xr, xvoff + mt * 1024, (int)(kstep(i) * 4096), AUX_SYS);
      SCHED();
    }
  };

  h8 wf[8][2];
#pragma unroll
  for (int i = 0; i < 8; ++i) {
    u4 f0, f1;
#pragma unroll
    for (int h = 0; h < 2; ++h) {
      const f4 r0 = raw[4 * i + 2 * h], r1 = raw[4 * i + 2 * h + 1];
      const unsigned p0 = cvt2(r0.x, r1.x), p1 = cvt2(r0.y, r1.y), p2 = cvt2(r0.z, r1.z), p3 = cvt2(r0.w, r1.w);
      f0[2 * h] = dpp_hi(p0, p2); f0[2 * h + 1] = dpp_lo(p2, p0);
      f1[2 * h] = dpp_hi(p1, p3); f1[2 * h + 1] = dpp_lo(p3, p1);
    }
    wf[i][0] = __builtin_bit_cast(h8, f0);
    wf[i][1] = __builtin_bit_cast(h8, f1);
    asm volatile("" : "+v"(wf[i][0]), "+v"(wf[i][1]));
    if (i < 5) {
#pragma unroll
      for (int k = 0; k < 4; ++k) raw[12 + 4 * i + k] = wload(kstep(3 + i), k);
    } else {
      xreq(i - 5);
    }
    SCHED();
  }

  f4 acc[4][2];
#pragma unroll
  for (int mt = 0; mt < 4; ++mt) { acc[mt][0] = (f4)0.f; acc[mt][1] = (f4)0.f; }
  int wa[4], ra, ovoff, osoff;
  auto put2 = [&](int mt) {
#pragma unroll
    for (int e = 0; e < 4; ++e) {
      *(__attribute__((address_space(3))) f2*)((lchar*)red + wa[e] + mt * 2048) = f2{acc[mt][0][e], acc[mt][1][e]};
      *(__attribute__((address_space(3))) f2*)((lchar*)red + wa[e] + mt * 2048 + 2048) = f2{acc[mt + 1][0][e], acc[mt + 1][1][e]};
    }
  };
#pragma unroll
  for (int i = 0; i < 8; ++i) {
    if (__builtin_expect(any_zero(xf[i]) && !((trusted >> i) & 1), 0)) {
      do {
        await(i);
#pragma unroll
        for (int e = i; e < i + 3 && e < 8; ++e) {
          if (e > i && !((seen >> e) & 1)) await(e);
          xreq(e);
        }
      } while (any_zero(xf[i]) && !((trusted >> i) & 1));
    }
    if (i + 3 < 8) {
      if (__builtin_expect(!((seen >> (i + 3)) & 1), 0)) await(i + 3);
      xreq(i + 3);
    }
    if (i == 7 && __builtin_expect(timed_out, 0)) {
#pragma unroll
      for (int mt = 0; mt < 4; ++mt) { acc[mt][0] = (f4)__builtin_nanf(""); acc[mt][1] = acc[mt][0]; }
    }
#pragma unroll
    for (int mt = 0; mt < 4; ++mt) {
      acc[mt][0] = __builtin_amdgcn_mfma_f32_16x16x32_f16(wf[i][0], __builtin_bit_cast(h8, xf[i][mt]), acc[mt][0], 0, 0, 0);
      acc[mt][1] = __builtin_amdgcn_mfma_f32_16x16x32_f16(wf[i][1], __builtin_bit_cast(h8, xf[i][mt]), acc[mt][1], 0, 0, 0);
    }
    if (i == 7) {
      put2(0);
      __builtin_amdgcn_sched_group_barrier(0x8, 5, 0);
#pragma unroll
      for (int k = 0; k < 3; ++k) {
        __builtin_amdgcn_sched_group_barrier(0x2, 4, 0);
        __builtin_amdgcn_sched_group_barrier(0x200, 1, 0);
        __builtin_amdgcn_sched_group_barrier(0x8, 1, 0);
      }
      SCHED();
      put2(2);
    }
    if (i == 5) {
      const int wb = (int)p * 8192 + c16 * 128 + 8 * ((q >> 1) ^ (c16 >> 3));
#pragma unroll
      for (int e = 0; e < 4; ++e) { wa[e] = wb + (((4 * (q & 1) + e) ^ (c16 & 7)) * 16); asm volatile("" : "+v"(wa[e])); }
      const int row = 8 * (int)p + (lane >> 3), seg = lane & 7;
      ra = row * 128 + ((seg ^ (row & 7)) * 16);
      ovoff = (row * ND + 4 * seg) * 4;
      osoff = (int)((64 * m * ND + n0) * 4);
      asm volatile("" : "+v"(ra), "+v"(ovoff));
      asm volatile("" : "+s"(osoff));
    }
    SCHED();
  }
  __amdgpu_buffer_rsrc_t orr = __builtin_amdgcn_make_buffer_rsrc((void*)outp, 0, MS * ND * 4, 0x00020000);
  __syncthreads();
  {
    f4 v[8];
#pragma unroll
    for (int w = 0; w < 8; ++w) v[w] = *(const __attribute__((address_space(3))) f4*)((lchar*)red + ra + w * 8192);
    SCHED();
    f4 sum = ((v[0] + v[1]) + (v[2] + v[3])) + ((v[4] + v[5]) + (v[6] + v[7]));
    if (p & 1) sum = sum.zwxy;
    f4 o;
#pragma unroll
    for (int e = 0; e < 4; ++e) o[e] = __builtin_elementwise_maximum(sum[e], 0.0f);
    __builtin_amdgcn_raw_buffer_store_b128(__builtin_bit_cast(u4, o), orr, ovoff, osoff, AUX_WT);
  }
  if (tid == 64) __hip_atomic_store((gu32*)(myws + CNT_OFF + b * 128), call, __ATOMIC_RELAXED, __HIP_MEMORY_SCOPE_AGENT);
}

extern "C" void kernel_launch(int rank, int nranks, ncclComm_t comm, hipStream_t stream,
                              void* const* d_in, const int* in_sizes, int n_in,
                              void* d_out, int out_size,
                              const int* in_dtypes, int out_dtype,
                              void* const* d_out_all,
                              void* const* d_ws_all, size_t ws_bytes) {
  if (nranks != NR || (unsigned)rank >= (unsigned)NR || n_in < 2 || ws_bytes < WS_NEEDED ||
      in_sizes[0] != KD * KS || in_sizes[1] != KD * ND || out_size != MS * ND) {
    fprintf(stderr, "kernel_launch: unexpected shapes\n");
    return;
  }
  Args a;
  a.x = (const float*)d_in[0];
  a.w = (const float*)d_in[1];
  a.out = (float*)d_out;
  a.myws = (char*)d_ws_all[rank];
  a.rank = rank;
  a.pad = 0;
  for (int i = 0; i < NR; ++i) a.wsrot[i] = (char*)d_ws_all[(rank + 1 + i) & 7];
  hipLaunchKernelGGL(a2a_gemm_relu, dim3(NWG), dim3(NTHREADS), 0, stream, a);
  HIP_CHECK(hipGetLastError());
}
```

```cpp
#include <hip/hip_runtime.h>
#include <rccl/rccl.h>
#include <cstddef>
#include <cstdio>

#define HIP_CHECK(expr) do { hipError_t _e = (expr); if (_e != hipSuccess) { \
    fprintf(stderr, "HIP error %s at %s:%d\n", hipGetErrorString(_e), __FILE__, __LINE__); \
    return; } } while (0)

typedef _Float16 h2 __attribute__((ext_vector_type(2)));
typedef _Float16 h8 __attribute__((ext_vector_type(8)));
typedef float f2 __attribute__((ext_vector_type(2)));
typedef float f4 __attribute__((ext_vector_type(4)));
typedef unsigned u4 __attribute__((ext_vector_type(4)));
typedef unsigned u8v __attribute__((ext_vector_type(8)));
typedef __attribute__((address_space(1))) unsigned gu32;
typedef __attribute__((address_space(4))) const unsigned cu32;
typedef __attribute__((address_space(3))) char lchar;

constexpr int NR = 8;
constexpr int KD = 2048, ND = 2048;
constexpr int MS = 256;
constexpr int KS = 256;
constexpr int NWG = 256;
constexpr int NTHREADS = 576;
constexpr int AMIN = 3, ASPAN = 6;
constexpr unsigned RESPERM = 0x64201573u;
constexpr int AUX_SYS = 17, AUX_WT = 16;

constexpr unsigned CNT_OFF = 0;
constexpr unsigned FLAG_OFF = 32768;
constexpr unsigned TMO_OFF = 40960;
constexpr unsigned XG_OFF = 1u << 20;
constexpr unsigned XG_BYTES = 1u << 20;
constexpr unsigned WS_NEEDED = XG_OFF + 3 * XG_BYTES;
constexpr unsigned long long DEADLINE_TICKS = 50000000ull;

struct Args {
  const float* x;
  const float* w;
  float* out;
  char* myws;
  int rank;
  int pad;
  char* wsrot[NR];
};
static_assert(offsetof(Args, x) == 0 && offsetof(Args, w) == 8 && offsetof(Args, out) == 16 && offsetof(Args, myws) == 24 &&
              offsetof(Args, rank) == 32, "the kernel reads these by offset");

#define INLINE __attribute__((always_inline))
#define SCHED() __builtin_amdgcn_sched_barrier(0)
#define BAR() __builtin_amdgcn_s_barrier()

__device__ __forceinline__ unsigned cvt2(float a, float b) {
  f2 v = {a, b};
  return __builtin_bit_cast(unsigned, __builtin_convertvector(v, h2));
}
__device__ __forceinline__ unsigned dpp_hi(unsigned old, unsigned src) { return __builtin_amdgcn_update_dpp(old, src, 0x128, 0xf, 0xc, false); }
__device__ __forceinline__ unsigned dpp_lo(unsigned old, unsigned src) { return __builtin_amdgcn_update_dpp(old, src, 0x128, 0xf, 0x3, false); }
__device__ __forceinline__ bool any_zero(const u4* f) {
  unsigned mn = min(min(f[0].x, f[0].y), f[0].z);
  mn = min(min(mn, f[0].w), f[1].x); mn = min(min(mn, f[1].y), f[1].z);
  mn = min(min(mn, f[1].w), f[2].x); mn = min(min(mn, f[2].y), f[2].z);
  mn = min(min(mn, f[2].w), f[3].x); mn = min(min(mn, f[3].y), f[3].z);
  mn = min(mn, f[3].w);
  return __builtin_amdgcn_ballot_w64(mn == 0u) != 0ull;
}

__global__ __launch_bounds__(NTHREADS, 3) void a2a_gemm_relu(Args a) {
  __shared__ __attribute__((aligned(16))) char red[8 * 8192];
  u8v ka;
  unsigned rank;
  {
    const auto kp = __builtin_amdgcn_kernarg_segment_ptr();
    asm volatile("s_load_dwordx8 %0, %2, 0x0\n\ts_load_dword %1, %2, 0x20" : "=&s"(ka), "=&s"(rank) : "s"(kp));
    SCHED();
  }
  const int tid = threadIdx.x;
  const int lane = tid & 63;
  const int wave = __builtin_amdgcn_readfirstlane(tid >> 6);
  const unsigned b = blockIdx.x;
  const int c16 = lane & 15, q = lane >> 4;
  const unsigned xl = b & 7, j = b >> 3;
  const unsigned m = j & 3, ns = j >> 2;
  const unsigned n0 = 32 * ((RESPERM >> (4 * xl)) & 7) + 256 * ns;
  const bool comm = wave == 0;
  const unsigned p = (unsigned)(wave - 1) & 7;

  auto kstep = [&](int i) INLINE -> unsigned { return (i + 2 * m) & 7; };
  unsigned dr7 = (xl - 1 + j) % 7;
  asm volatile("" : "+s"(dr7));
  const unsigned d = xl == 0 ? 7u : dr7;
  const unsigned us = (ns + 2 * m) & 7;
  const int wvoff = ((8 * q + 2 * ((lane >> 3) & 1)) * ND + 4 * (lane & 7)) * 4;
  const int wbase = (KS * p * ND + n0) * 4;

  int xb0 = (int)((64 * m * KS + 32 * us) * 4), wb0 = wbase + (int)kstep(0) * 32 * ND * 4, wb1 = wbase + (int)kstep(1) * 32 * ND * 4;
  unsigned ap = AMIN + (j * ASPAN) / 32;
  asm volatile("" : "+s"(xb0), "+s"(wb0), "+s"(wb1), "+s"(ap));
  int fvoff = comm ? (c16 * KS + 8 * q) * 4 : wvoff;
  const int fb0 = comm ? xb0 : wb0, fb1 = comm ? xb0 + 32 * KS * 4 : wb1;
  const int fsa = comm ? 16 : ND * 4, fsb = comm ? 16 * KS * 4 : 4 * ND * 4;
  int fso[8];
#pragma unroll
  for (int n = 0; n < 8; ++n) fso[n] = (n < 4 ? fb0 : fb1) + (n & 1) * fsa + ((n >> 1) & 1) * fsb;
  unsigned frk = comm ? MS * KS * 4 : 0, fbytes = comm ? MS * KS * 4 : KD * ND * 4, dp1 = d + 1, cnto = CNT_OFF + b * 128;
  unsigned bpos = comm ? 99u : ap;
  asm volatile("" : "+v"(fvoff));
  asm volatile("" : "+s"(fso[0]), "+s"(fso[1]), "+s"(fso[2]), "+s"(fso[3]), "+s"(fso[4]), "+s"(fso[5]), "+s"(fso[6]), "+s"(fso[7]));
  asm volatile("" : "+s"(frk), "+s"(fbytes), "+s"(bpos), "+s"(dp1), "+s"(cnto));
  asm volatile("s_waitcnt lgkmcnt(0)" : "+s"(ka), "+s"(rank));
  const float* xp = (const float*)(((unsigned long long)ka[1] << 32) | ka[0]);
  const float* wp = (const float*)(((unsigned long long)ka[3] << 32) | ka[2]);
  float* outp = (float*)(((unsigned long long)ka[5] << 32) | ka[4]);
  char* myws = (char*)(((unsigned long long)ka[7] << 32) | ka[6]);
  f4 raw[32];
  unsigned cntv;
  {
    const char* fbase = (const char*)(comm ? xp : wp) + ((rank + dp1) & 7) * frk;
    asm volatile("" : "+s"(fbase));
    __amdgpu_buffer_rsrc_t fr = __builtin_amdgcn_make_buffer_rsrc((void*)fbase, 0, fbytes, 0x00020000);
    SCHED();
    if (!comm) BAR();
#pragma unroll
    for (int n = 0; n < 8; ++n) {
      if (n >= AMIN && n == (int)bpos) { BAR(); BAR(); }
      raw[n] = __builtin_bit_cast(f4, __builtin_amdgcn_raw_buffer_load_b128(fr, fvoff, fso[n], 0));
      if (n == 2) cntv = *(cu32*)(myws + cnto);
      SCHED();
    }
  }

  if (__builtin_expect(comm, 1)) {
    BAR();
    SCHED();
    __builtin_amdgcn_s_setprio(3);
    char* dws = a.wsrot[d];
    unsigned cv = cntv;
    asm volatile("" : "+s"(dws), "+s"(cv));
    SCHED();
    const unsigned call = cv + 1;
    __amdgpu_buffer_rsrc_t dr = __builtin_amdgcn_make_buffer_rsrc((void*)(dws + XG_OFF + (call % 3) * XG_BYTES), 0, XG_BYTES, 0x00020000);
    __amdgpu_buffer_rsrc_t zr = __builtin_amdgcn_make_buffer_rsrc((void*)(myws + XG_OFF + ((call + 2) % 3) * XG_BYTES), 0, XG_BYTES, 0x00020000);
    const int dbase = (int)(((rank * 4 + m) * 8 + us) * 4096);
    u4 h[4];
#pragma unroll
    for (int t = 0; t < 4; ++t) {
      const f4 lo = raw[2 * t], hi = raw[2 * t + 1];
      h[t] = u4{cvt2(lo.x, lo.y), cvt2(lo.z, lo.w), cvt2(hi.x, hi.y), cvt2(hi.z, hi.w)};
      asm volatile("" : "+v"(h[t]));
    }
    SCHED();
    BAR();
#pragma unroll
    for (int t = 0; t < 4; ++t) __builtin_amdgcn_raw_buffer_store_b128(h[t], dr, lane * 16 + t * 1024, dbase, AUX_SYS);
#pragma unroll
    for (int t = 0; t < 4; ++t) __builtin_amdgcn_raw_buffer_store_b128(u4{0u, 0u, 0u, 0u}, zr, lane * 16 + t * 1024, (int)(b * 4096), AUX_WT);
    SCHED();
    BAR();
    BAR();
    if (any_zero(h)) {
      asm volatile("s_waitcnt vmcnt(0)" ::: "memory");
      if (lane == 0)
        __hip_atomic_store((gu32*)(dws + FLAG_OFF + (rank * 4 + m) * 128 + ns * 4), call, __ATOMIC_RELAXED, __HIP_MEMORY_SCOPE_SYSTEM);
    }
    asm volatile("s_endpgm" ::: "memory");
  }

  __amdgpu_buffer_rsrc_t wr = __builtin_amdgcn_make_buffer_rsrc((void*)wp, 0, KD * ND * 4, 0x00020000);
  auto wload = [&](unsigned s, int k) INLINE -> f4 {
    const int soff = wbase + (int)(s * 32 + (k & 1) + 4 * (k >> 1)) * ND * 4;
    return __builtin_bit_cast(f4, __builtin_amdgcn_raw_buffer_load_b128(wr, wvoff, soff, 0));
  };
  if (bpos == 8) { BAR(); BAR(); }
  BAR();
#pragma unroll
  for (int n = 8; n < 12; ++n) raw[n] = wload(kstep(2), n & 3);
  SCHED();
  asm volatile("" : "+s"(cntv));
  const unsigned call = cntv + 1;
  __amdgpu_buffer_rsrc_t xr = __builtin_amdgcn_make_buffer_rsrc((void*)(myws + XG_OFF + (call % 3) * XG_BYTES + (p * 4 + m) * 32768), 0, 32768, 0x00020000);
  const int xvoff = lane * 16;

  unsigned seen = 255u, flagged = 0, trusted = 0;
  bool timed_out = false;
  const unsigned long long t0 = __builtin_amdgcn_s_memrealtime();
  auto await = [&](int i) INLINE {
    const int pvoff = lane < 8 ? (int)(((lane + 2 * m) & 7) * 4096 + 3 * 1024 + xl * 128 + 124) : 0x7fffffff;
    gu32* fl = (gu32*)(myws + FLAG_OFF + (p * 4 + m) * 128) + (lane & 7);
    gu32* tmo = (gu32*)(myws + TMO_OFF);
    for (;;) {
      asm volatile("" ::: "memory");
      const unsigned v = __builtin_amdgcn_raw_buffer_load_b32(xr, pvoff, 0, AUX_SYS);
      const unsigned f = __hip_atomic_load(fl, __ATOMIC_RELAXED, __HIP_MEMORY_SCOPE_SYSTEM);
      const unsigned t = __hip_atomic_load(tmo, __ATOMIC_RELAXED, __HIP_MEMORY_SCOPE_AGENT);
      flagged = (unsigned)__builtin_amdgcn_ballot_w64((int)(f - call) >= 0) & 255u;
      seen = (unsigned)__builtin_amdgcn_ballot_w64(v != 0u) | flagged;
      if (t == call || __builtin_amdgcn_s_memrealtime() - t0 > DEADLINE_TICKS) {
        __hip_atomic_store(tmo, call, __ATOMIC_RELAXED, __HIP_MEMORY_SCOPE_AGENT);
        timed_out = true;
        seen = flagged = trusted = 255u;
      }
      if ((seen >> i) & 1) break;
      __builtin_amdgcn_s_sleep(1);
    }
  };
  u4 xf[8][4];
  auto xreq = [&](int i) INLINE {
    trusted |= flagged & (1u << i);
    asm volatile("" ::: "memory");
    SCHED();
#pragma unroll
    for (int mt = 0; mt < 4; ++mt) {
      xf[i][mt] = __builtin_amdgcn_raw_buffer_load_b128(xr, xvoff + mt * 1024, (int)(kstep(i) * 4096), AUX_SYS);
      SCHED();
    }
  };

  h8 wf[8][2];
#pragma unroll
  for (int i = 0; i < 8; ++i) {
    u4 f0, f1;
#pragma unroll
    for (int h = 0; h < 2; ++h) {
      const f4 r0 = raw[4 * i + 2 * h], r1 = raw[4 * i + 2 * h + 1];
      const unsigned p0 = cvt2(r0.x, r1.x), p1 = cvt2(r0.y, r1.y), p2 = cvt2(r0.z, r1.z), p3 = cvt2(r0.w, r1.w);
      f0[2 * h] = dpp_hi(p0, p2); f0[2 * h + 1] = dpp_lo(p2, p0);
      f1[2 * h] = dpp_hi(p1, p3); f1[2 * h + 1] = dpp_lo(p3, p1);
    }
    wf[i][0] = __builtin_bit_cast(h8, f0);
    wf[i][1] = __builtin_bit_cast(h8, f1);
    asm volatile("" : "+v"(wf[i][0]), "+v"(wf[i][1]));
    if (i < 5) {
#pragma unroll
      for (int k = 0; k < 4; ++k) raw[12 + 4 * i + k] = wload(kstep(3 + i), k);
    } else {
      xreq(i - 5);
    }
    SCHED();
  }

  f4 acc[4][2];
#pragma unroll
  for (int mt = 0; mt < 4; ++mt) { acc[mt][0] = (f4)0.f; acc[mt][1] = (f4)0.f; }
  int wa[4], ra, ovoff, osoff;
  auto put2 = [&](int mt) INLINE {
#pragma unroll
    for (int e = 0; e < 4; ++e) {
      *(__attribute__((address_space(3))) f2*)((lchar*)red + wa[e] + mt * 2048) = f2{acc[mt][0][e], acc[mt][1][e]};
      *(__attribute__((address_space(3))) f2*)((lchar*)red + wa[e] + mt * 2048 + 2048) = f2{acc[mt + 1][0][e], acc[mt + 1][1][e]};
    }
  };
#pragma unroll
  for (int i = 0; i < 8; ++i) {
    if (__builtin_expect(any_zero(xf[i]) && !((trusted >> i) & 1), 0)) {
      do {
        await(i);
#pragma unroll
        for (int e = i; e < i + 3 && e < 8; ++e) {
          if (e > i && !((seen >> e) & 1)) await(e);
          xreq(e);
        }
      } while (any_zero(xf[i]) && !((trusted >> i) & 1));
    }
    if (i + 3 < 8) {
      if (__builtin_expect(!((seen >> (i + 3)) & 1), 0)) await(i + 3);
      xreq(i + 3);
    }
    if (i == 7 && __builtin_expect(timed_out, 0)) {
#pragma unroll
      for (int mt = 0; mt < 4; ++mt) { acc[mt][0] = (f4)__builtin_nanf(""); acc[mt][1] = acc[mt][0]; }
    }
#pragma unroll
    for (int mt = 0; mt < 4; ++mt) {
      acc[mt][0] = __builtin_amdgcn_mfma_f32_16x16x32_f16(wf[i][0], __builtin_bit_cast(h8, xf[i][mt]), acc[mt][0], 0, 0, 0);
      acc[mt][1] = __builtin_amdgcn_mfma_f32_16x16x32_f16(wf[i][1], __builtin_bit_cast(h8, xf[i][mt]), acc[mt][1], 0, 0, 0);
    }
    if (i == 7) {
      put2(0);
      __builtin_amdgcn_sched_group_barrier(0x8, 5, 0);
#pragma unroll
      for (int k = 0; k < 3; ++k) {
        __builtin_amdgcn_sched_group_barrier(0x2, 4, 0);
        __builtin_amdgcn_sched_group_barrier(0x200, 1, 0);
        __builtin_amdgcn_sched_group_barrier(0x8, 1, 0);
      }
      SCHED();
      put2(2);
    }
    if (i == 5) {
      const int wb = (int)p * 8192 + c16 * 128 + 8 * ((q >> 1) ^ (c16 >> 3));
#pragma unroll
      for (int e = 0; e < 4; ++e) { wa[e] = wb + (((4 * (q & 1) + e) ^ (c16 & 7)) * 16); asm volatile("" : "+v"(wa[e])); }
      const int row = 8 * (int)p + (lane >> 3), seg = lane & 7;
      ra = row * 128 + ((seg ^ (row & 7)) * 16);
      ovoff = (row * ND + 4 * seg) * 4;
      osoff = (int)((64 * m * ND + n0) * 4);
      asm volatile("" : "+v"(ra), "+v"(ovoff));
      asm volatile("" : "+s"(osoff));
    }
    SCHED();
  }
  __amdgpu_buffer_rsrc_t orr = __builtin_amdgcn_make_buffer_rsrc((void*)outp, 0, MS * ND * 4, 0x00020000);
  __syncthreads();
  {
    f4 v[8];
#pragma unroll
    for (int w = 0; w < 8; ++w) v[w] = *(const __attribute__((address_space(3))) f4*)((lchar*)red + ra + w * 8192);
    SCHED();
    f4 sum = ((v[0] + v[1]) + (v[2] + v[3])) + ((v[4] + v[5]) + (v[6] + v[7]));
    if (p & 1) sum = sum.zwxy;
    f4 o;
#pragma unroll
    for (int e = 0; e < 4; ++e) o[e] = __builtin_elementwise_maximum(sum[e], 0.0f);
    __builtin_amdgcn_raw_buffer_store_b128(__builtin_bit_cast(u4, o), orr, ovoff, osoff, AUX_WT);
  }
  if (tid == 64) __hip_atomic_store((gu32*)(myws + CNT_OFF + b * 128), call, __ATOMIC_RELAXED, __HIP_MEMORY_SCOPE_AGENT);
}

extern "C" void kernel_launch(int rank, int nranks, ncclComm_t comm, hipStream_t stream,
                              void* const* d_in, const int* in_sizes, int n_in,
                              void* d_out, int out_size,
                              const int* in_dtypes, int out_dtype,
                              void* const* d_out_all,
                              void* const* d_ws_all, size_t ws_bytes) {
  if (nranks != NR || (unsigned)rank >= (unsigned)NR || n_in < 2 || ws_bytes < WS_NEEDED ||
      in_sizes[0] != KD * KS || in_sizes[1] != KD * ND || out_size != MS * ND) {
    fprintf(stderr, "kernel_launch: unexpected shapes\n");
    return;
  }
  Args a;
  a.x = (const float*)d_in[0];
  a.w = (const float*)d_in[1];
  a.out = (float*)d_out;
  a.myws = (char*)d_ws_all[rank];
  a.rank = rank;
  a.pad = 0;
  for (int i = 0; i < NR; ++i) a.wsrot[i] = (char*)d_ws_all[(rank + 1 + i) & 7];
  hipLaunchKernelGGL(a2a_gemm_relu, dim3(NWG), dim3(NTHREADS), 0, stream, a);
  HIP_CHECK(hipGetLastError());
}
```

```cpp
#include <hip/hip_runtime.h>
#include <rccl/rccl.h>
#include <cstddef>
#include <cstdio>

#define HIP_CHECK(expr) do { hipError_t _e = (expr); if (_e != hipSuccess) { \
    fprintf(stderr, "HIP error %s at %s:%d\n", hipGetErrorString(_e), __FILE__, __LINE__); \
    return; } } while (0)

typedef _Float16 h2 __attribute__((ext_vector_type(2)));
typedef _Float16 h8 __attribute__((ext_vector_type(8)));
typedef float f2 __attribute__((ext_vector_type(2)));
typedef float f4 __attribute__((ext_vector_type(4)));
typedef unsigned u4 __attribute__((ext_vector_type(4)));
typedef unsigned u8v __attribute__((ext_vector_type(8)));
typedef __attribute__((address_space(1))) unsigned gu32;
typedef __attribute__((address_space(4))) const unsigned cu32;
typedef __attribute__((address_space(3))) char lchar;

constexpr int NR = 8;
constexpr int KD = 2048, ND = 2048;
constexpr int MS = 256;
constexpr int KS = 256;
constexpr int NWG = 256;
constexpr int NTHREADS = 576;
constexpr int AMIN = 3, ASPAN = 6;
constexpr unsigned RESPERM = 0x64201573u;
constexpr int AUX_SYS = 17, AUX_WT = 16;

constexpr unsigned CNT_OFF = 0;
constexpr unsigned FLAG_OFF = 32768;
constexpr unsigned TMO_OFF = 40960;
constexpr unsigned XG_OFF = 1u << 20;
constexpr unsigned XG_BYTES = 1u << 20;
constexpr unsigned WS_NEEDED = XG_OFF + 3 * XG_BYTES;
constexpr unsigned long long DEADLINE_TICKS = 50000000ull;

struct Args {
  const float* x;
  const float* w;
  float* out;
  char* myws;
  int rank;
  int pad;
  char* wsrot[NR];
};
static_assert(offsetof(Args, x) == 0 && offsetof(Args, w) == 8 && offsetof(Args, out) == 16 && offsetof(Args, myws) == 24 &&
              offsetof(Args, rank) == 32, "the kernel reads these by offset");

#define INLINE __attribute__((always_inline))
#define SCHED() __builtin_amdgcn_sched_barrier(0)
#define BAR() __builtin_amdgcn_s_barrier()

__device__ __forceinline__ unsigned cvt2(float a, float b) {
  f2 v = {a, b};
  return __builtin_bit_cast(unsigned, __builtin_convertvector(v, h2));
}
__device__ __forceinline__ unsigned dpp_hi(unsigned old, unsigned src) { return __builtin_amdgcn_update_dpp(old, src, 0x128, 0xf, 0xc, false); }
__device__ __forceinline__ unsigned dpp_lo(unsigned old, unsigned src) { return __builtin_amdgcn_update_dpp(old, src, 0x128, 0xf, 0x3, false); }
__device__ __forceinline__ bool any_zero(const u4* f) {
  unsigned mn = min(min(f[0].x, f[0].y), f[0].z);
  mn = min(min(mn, f[0].w), f[1].x); mn = min(min(mn, f[1].y), f[1].z);
  mn = min(min(mn, f[1].w), f[2].x); mn = min(min(mn, f[2].y), f[2].z);
  mn = min(min(mn, f[2].w), f[3].x); mn = min(min(mn, f[3].y), f[3].z);
  mn = min(mn, f[3].w);
  return __builtin_amdgcn_ballot_w64(mn == 0u) != 0ull;
}

__global__ __launch_bounds__(NTHREADS, 3) void a2a_gemm_relu(Args a) {
  __shared__ __attribute__((aligned(16))) char red[8 * 8192];
  u8v ka;
  unsigned rank;
  {
    const auto kp = __builtin_amdgcn_kernarg_segment_ptr();
    asm volatile("s_load_dwordx8 %0, %2, 0x0\n\ts_load_dword %1, %2, 0x20" : "=&s"(ka), "=&s"(rank) : "s"(kp));
    SCHED();
  }
  const int tid = threadIdx.x;
  const int lane = tid & 63;
  const int wave = __builtin_amdgcn_readfirstlane(tid >> 6);
  const unsigned b = blockIdx.x;
  const int c16 = lane & 15, q = lane >> 4;
  const unsigned xl = b & 7, j = b >> 3;
  const unsigned m = j & 3, ns = j >> 2;
  const unsigned n0 = 32 * ((RESPERM >> (4 * xl)) & 7) + 256 * ns;
  const bool comm = wave == 0;
  const unsigned p = (unsigned)(wave - 1) & 7;

  auto kstep = [&](int i) INLINE -> unsigned { return (i + 2 * m) & 7; };
  unsigned dr7 = (xl - 1 + j) % 7;
  asm volatile("" : "+s"(dr7));
  const unsigned d = xl == 0 ? 7u : dr7;
  const unsigned us = (ns + 2 * m) & 7;
  const int wvoff = ((8 * q + 2 * ((lane >> 3) & 1)) * ND + 4 * (lane & 7)) * 4;
  const int wbase = (KS * p * ND + n0) * 4;

  int xb0 = (int)((64 * m * KS + 32 * us) * 4), wb0 = wbase + (int)kstep(0) * 32 * ND * 4, wb1 = wbase + (int)kstep(1) * 32 * ND * 4;
  unsigned ap = AMIN + (j * ASPAN) / 32;
  asm volatile("" : "+s"(xb0), "+s"(wb0), "+s"(wb1), "+s"(ap));
  int fvoff = comm ? (c16 * KS + 8 * q) * 4 : wvoff;
  const int fb0 = comm ? xb0 : wb0, fb1 = comm ? xb0 + 32 * KS * 4 : wb1;
  const int fsa = comm ? 16 : ND * 4, fsb = comm ? 16 * KS * 4 : 4 * ND * 4;
  int fso[8];
#pragma unroll
  for (int n = 0; n < 8; ++n) fso[n] = (n < 4 ? fb0 : fb1) + (n & 1) * fsa + ((n >> 1) & 1) * fsb;
  unsigned frk = comm ? MS * KS * 4 : 0, fbytes = comm ? MS * KS * 4 : KD * ND * 4, dp1 = d + 1, cnto = CNT_OFF + b * 128;
  unsigned bpos = comm ? 99u : ap;
  asm volatile("" : "+v"(fvoff));
  asm volatile("" : "+s"(fso[0]), "+s"(fso[1]), "+s"(fso[2]), "+s"(fso[3]), "+s"(fso[4]), "+s"(fso[5]), "+s"(fso[6]), "+s"(fso[7]));
  asm volatile("" : "+s"(frk), "+s"(fbytes), "+s"(bpos), "+s"(dp1), "+s"(cnto));
  asm volatile("s_waitcnt lgkmcnt(0)" : "+s"(ka), "+s"(rank));
  const float* xp = (const float*)(((unsigned long long)ka[1] << 32) | ka[0]);
  const float* wp = (const float*)(((unsigned long long)ka[3] << 32) | ka[2]);
  float* outp = (float*)(((unsigned long long)ka[5] << 32) | ka[4]);
  char* myws = (char*)(((unsigned long long)ka[7] << 32) | ka[6]);
  f4 raw[32];
  unsigned cntv;
  {
    const char* fbase = (const char*)(comm ? xp : wp) + ((rank + dp1) & 7) * frk;
    asm volatile("" : "+s"(fbase));
    __amdgpu_buffer_rsrc_t fr = __builtin_amdgcn_make_buffer_rsrc((void*)fbase, 0, fbytes, 0x00020000);
    SCHED();
    if (!comm) BAR();
#pragma unroll
    for (int n = 0; n < 8; ++n) {
      if (n >= AMIN && n == (int)bpos) { BAR(); BAR(); }
      raw[n] = __builtin_bit_cast(f4, __builtin_amdgcn_raw_buffer_load_b128(fr, fvoff, fso[n], 0));
      if (n == 2) cntv = *(cu32*)(myws + cnto);
      SCHED();
    }
  }

  if (__builtin_expect(comm, 1)) {
    BAR();
    SCHED();
    __builtin_amdgcn_s_setprio(3);
    char* dws = a.wsrot[d & 7];
    unsigned cv = cntv;
    asm volatile("" : "+s"(dws), "+s"(cv));
    SCHED();
    const unsigned call = cv + 1;
    __amdgpu_buffer_rsrc_t dr = __builtin_amdgcn_make_buffer_rsrc((void*)(dws + XG_OFF + (call % 3) * XG_BYTES), 0, XG_BYTES, 0x00020000);
    __amdgpu_buffer_rsrc_t zr = __builtin_amdgcn_make_buffer_rsrc((void*)(myws + XG_OFF + ((call + 2) % 3) * XG_BYTES), 0, XG_BYTES, 0x00020000);
    const int dbase = (int)(((rank * 4 + m) * 8 + us) * 4096);
    u4 h[4];
#pragma unroll
    for (int t = 0; t < 4; ++t) {
      const f4 lo = raw[2 * t], hi = raw[2 * t + 1];
      h[t] = u4{cvt2(lo.x, lo.y), cvt2(lo.z, lo.w), cvt2(hi.x, hi.y), cvt2(hi.z, hi.w)};
      asm volatile("" : "+v"(h[t]));
    }
    SCHED();
    BAR();
#pragma unroll
    for (int t = 0; t < 4; ++t) __builtin_amdgcn_raw_buffer_store_b128(h[t], dr, lane * 16 + t * 1024, dbase, AUX_SYS);
#pragma unroll
    for (int t = 0; t < 4; ++t) __builtin_amdgcn_raw_buffer_store_b128(u4{0u, 0u, 0u, 0u}, zr, lane * 16 + t * 1024, (int)(b * 4096), AUX_WT);
    SCHED();
    BAR();
    BAR();
    if (any_zero(h)) {
      asm volatile("s_waitcnt vmcnt(0)" ::: "memory");
      if (lane == 0)
        __hip_atomic_store((gu32*)(dws + FLAG_OFF + (rank * 4 + m) * 128 + ns * 4), call, __ATOMIC_RELAXED, __HIP_MEMORY_SCOPE_SYSTEM);
    }
    asm volatile("s_endpgm" ::: "memory");
  }

  __amdgpu_buffer_rsrc_t wr = __builtin_amdgcn_make_buffer_rsrc((void*)wp, 0, KD * ND * 4, 0x00020000);
  auto wload = [&](unsigned s, int k) INLINE -> f4 {
    const int soff = wbase + (int)(s * 32 + (k & 1) + 4 * (k >> 1)) * ND * 4;
    return __builtin_bit_cast(f4, __builtin_amdgcn_raw_buffer_load_b128(wr, wvoff, soff, 0));
  };
  if (bpos == 8) { BAR(); BAR(); }
  BAR();
#pragma unroll
  for (int n = 8; n < 12; ++n) raw[n] = wload(kstep(2), n & 3);
  SCHED();
  asm volatile("" : "+s"(cntv));
  const unsigned call = cntv + 1;
  __amdgpu_buffer_rsrc_t xr = __builtin_amdgcn_make_buffer_rsrc((void*)(myws + XG_OFF + (call % 3) * XG_BYTES + (p * 4 + m) * 32768), 0, 32768, 0x00020000);
  const int xvoff = lane * 16;

  unsigned seen = 0, flagged = 0, trusted = 0;
  bool timed_out = false;
  const unsigned long long t0 = __builtin_amdgcn_s_memrealtime();
  __amdgpu_buffer_rsrc_t pr = __builtin_amdgcn_make_buffer_rsrc((void*)myws, 0, WS_NEEDED, 0x00020000);
  int pxo = (int)(XG_OFF + (call % 3) * XG_BYTES + (p * 4 + m) * 32768 + 3072 + xl * 128 + 124), pfo = (int)(FLAG_OFF + (p * 4 + m) * 128), pm2 = 2 * m;
  asm volatile("" : "+s"(pxo), "+s"(pfo), "+s"(pm2));
  int pvoff = lane < 8 ? pxo + ((lane + pm2) & 7) * 4096 : lane < 16 ? pfo + (lane & 7) * 4 : (int)TMO_OFF;
  asm volatile("" : "+v"(pvoff));
  constexpr int NOWHERE = 0x7fffffff;
  u4 xf[8][4];
  auto fetch = [&](int i) INLINE {
    if (__builtin_expect(!((seen >> i) & 1), 0)) {
      for (;;) {
        asm volatile("" ::: "memory");
        const unsigned pv = __builtin_amdgcn_raw_buffer_load_b32(pr, (lane < 8 && (lane < i || lane >= i + 4)) ? NOWHERE : pvoff, 0, AUX_SYS);
        flagged = ((unsigned)__builtin_amdgcn_ballot_w64((int)(pv - call) >= 0) >> 8) & 255u;
        seen = ((unsigned)__builtin_amdgcn_ballot_w64(pv != 0u) & 255u) | flagged;
        if ((((unsigned)__builtin_amdgcn_ballot_w64(pv == call) >> 16) & 1) || __builtin_amdgcn_s_memrealtime() - t0 > DEADLINE_TICKS) {
          __hip_atomic_store((gu32*)(myws + TMO_OFF), call, __ATOMIC_RELAXED, __HIP_MEMORY_SCOPE_AGENT);
          timed_out = true;
          seen = flagged = trusted = 255u;
        }
        if ((seen >> i) & 1) break;
        if (i < 7) __builtin_amdgcn_s_sleep(12); else __builtin_amdgcn_s_sleep(1);
      }
    }
    trusted |= flagged & (1u << i);
    asm volatile("" ::: "memory");
    SCHED();
#pragma unroll
    for (int mt = 0; mt < 4; ++mt) {
      xf[i][mt] = __builtin_amdgcn_raw_buffer_load_b128(xr, xvoff + mt * 1024, (int)(kstep(i) * 4096), AUX_SYS);
      SCHED();
    }
  };
  unsigned first = 0;

  h8 wf[8][2];
#pragma unroll
  for (int i = 0; i < 8; ++i) {
    u4 f0, f1;
#pragma unroll
    for (int h = 0; h < 2; ++h) {
      const f4 r0 = raw[4 * i + 2 * h], r1 = raw[4 * i + 2 * h + 1];
      const unsigned p0 = cvt2(r0.x, r1.x), p1 = cvt2(r0.y, r1.y), p2 = cvt2(r0.z, r1.z), p3 = cvt2(r0.w, r1.w);
      f0[2 * h] = dpp_hi(p0, p2); f0[2 * h + 1] = dpp_lo(p2, p0);
      f1[2 * h] = dpp_hi(p1, p3); f1[2 * h + 1] = dpp_lo(p3, p1);
    }
    wf[i][0] = __builtin_bit_cast(h8, f0);
    wf[i][1] = __builtin_bit_cast(h8, f1);
    asm volatile("" : "+v"(wf[i][0]), "+v"(wf[i][1]));
    if (i == 3) { first = __builtin_amdgcn_raw_buffer_load_b32(pr, lane >= 3 ? NOWHERE : pvoff, 0, AUX_SYS); SCHED(); }
    if (i == 5) {
      const unsigned mk = (unsigned)__builtin_amdgcn_ballot_w64(first != 0u);
      seen = mk == 7u ? 255u : mk;
    }
    if (i < 5) {
#pragma unroll
      for (int k = 0; k < 4; ++k) raw[12 + 4 * i + k] = wload(kstep(3 + i), k);
    } else {
      fetch(i - 5);
    }
    SCHED();
  }

  f4 acc[4][2];
#pragma unroll
  for (int mt = 0; mt < 4; ++mt) { acc[mt][0] = (f4)0.f; acc[mt][1] = (f4)0.f; }
  int wa[4], ra, ovoff, osoff;
  auto put2 = [&](int mt) INLINE {
#pragma unroll
    for (int e = 0; e < 4; ++e) {
      *(__attribute__((address_space(3))) f2*)((lchar*)red + wa[e] + mt * 2048) = f2{acc[mt][0][e], acc[mt][1][e]};
      *(__attribute__((address_space(3))) f2*)((lchar*)red + wa[e] + mt * 2048 + 2048) = f2{acc[mt + 1][0][e], acc[mt + 1][1][e]};
    }
  };
#pragma unroll
  for (int i = 0; i < 8; ++i) {
    if (__builtin_expect(any_zero(xf[i]) && !((trusted >> i) & 1), 0)) {
      do {
        seen &= ~(1u << i);
#pragma unroll
        for (int e = i; e < i + 3 && e < 8; ++e) fetch(e);
      } while (any_zero(xf[i]) && !((trusted >> i) & 1));
    }
    if (i + 3 < 8) fetch(i + 3);
    if (i == 7 && __builtin_expect(timed_out, 0)) {
#pragma unroll
      for (int mt = 0; mt < 4; ++mt) { acc[mt][0] = (f4)__builtin_nanf(""); acc[mt][1] = acc[mt][0]; }
    }
#pragma unroll
    for (int mt = 0; mt < 4; ++mt) {
      acc[mt][0] = __builtin_amdgcn_mfma_f32_16x16x32_f16(wf[i][0], __builtin_bit_cast(h8, xf[i][mt]), acc[mt][0], 0, 0, 0);
      acc[mt][1] = __builtin_amdgcn_mfma_f32_16x16x32_f16(wf[i][1], __builtin_bit_cast(h8, xf[i][mt]), acc[mt][1], 0, 0, 0);
    }
    if (i == 7) {
      put2(0);
      __builtin_amdgcn_sched_group_barrier(0x8, 5, 0);
#pragma unroll
      for (int k = 0; k < 3; ++k) {
        __builtin_amdgcn_sched_group_barrier(0x2, 4, 0);
        __builtin_amdgcn_sched_group_barrier(0x200, 1, 0);
        __builtin_amdgcn_sched_group_barrier(0x8, 1, 0);
      }
      SCHED();
      put2(2);
    }
    if (i == 5) {
      const int wb = (int)p * 8192 + c16 * 128 + 8 * ((q >> 1) ^ (c16 >> 3));
#pragma unroll
      for (int e = 0; e < 4; ++e) { wa[e] = wb + (((4 * (q & 1) + e) ^ (c16 & 7)) * 16); asm volatile("" : "+v"(wa[e])); }
      const int row = 8 * (int)p + (lane >> 3), seg = lane & 7;
      ra = row * 128 + ((seg ^ (row & 7)) * 16);
      ovoff = (row * ND + 4 * seg) * 4;
      osoff = (int)((64 * m * ND + n0) * 4);
      asm volatile("" : "+v"(ra), "+v"(ovoff));
      asm volatile("" : "+s"(osoff));
    }
    SCHED();
  }
  __amdgpu_buffer_rsrc_t orr = __builtin_amdgcn_make_buffer_rsrc((void*)outp, 0, MS * ND * 4, 0x00020000);
  __syncthreads();
  {
    f4 v[8];
#pragma unroll
    for (int w = 0; w < 8; ++w) v[w] = *(const __attribute__((address_space(3))) f4*)((lchar*)red + ra + w * 8192);
    SCHED();
    f4 sum = ((v[0] + v[1]) + (v[2] + v[3])) + ((v[4] + v[5]) + (v[6] + v[7]));
    if (p & 1) sum = sum.zwxy;
    f4 o;
#pragma unroll
    for (int e = 0; e < 4; ++e) o[e] = __builtin_elementwise_maximum(sum[e], 0.0f);
    __builtin_amdgcn_raw_buffer_store_b128(__builtin_bit_cast(u4, o), orr, ovoff, osoff, AUX_WT);
  }
  if (tid == 64) __hip_atomic_store((gu32*)(myws + CNT_OFF + b * 128), call, __ATOMIC_RELAXED, __HIP_MEMORY_SCOPE_AGENT);
}

extern "C" void kernel_launch(int rank, int nranks, ncclComm_t comm, hipStream_t stream,
                              void* const* d_in, const int* in_sizes, int n_in,
                              void* d_out, int out_size,
                              const int* in_dtypes, int out_dtype,
                              void* const* d_out_all,
                              void* const* d_ws_all, size_t ws_bytes) {
  if (nranks != NR || (unsigned)rank >= (unsigned)NR || n_in < 2 || ws_bytes < WS_NEEDED ||
      in_sizes[0] != KD * KS || in_sizes[1] != KD * ND || out_size != MS * ND) {
    fprintf(stderr, "kernel_launch: unexpected shapes\n");
    return;
  }
  Args a;
  a.x = (const float*)d_in[0];
  a.w = (const float*)d_in[1];
  a.out = (float*)d_out;
  a.myws = (char*)d_ws_all[rank];
  a.rank = rank;
  a.pad = 0;
  for (int i = 0; i < NR; ++i) a.wsrot[i] = (char*)d_ws_all[(rank + 1 + i) & 7];
  hipLaunchKernelGGL(a2a_gemm_relu, dim3(NWG), dim3(NTHREADS), 0, stream, a);
  HIP_CHECK(hipGetLastError());
}
```

```cpp
#include <hip/hip_runtime.h>
#include <rccl/rccl.h>
#include <cstddef>
#include <cstdio>

#define HIP_CHECK(expr) do { hipError_t _e = (expr); if (_e != hipSuccess) { \
    fprintf(stderr, "HIP error %s at %s:%d\n", hipGetErrorString(_e), __FILE__, __LINE__); \
    return; } } while (0)

typedef _Float16 h2 __attribute__((ext_vector_type(2)));
typedef _Float16 h8 __attribute__((ext_vector_type(8)));
typedef float f2 __attribute__((ext_vector_type(2)));
typedef float f4 __attribute__((ext_vector_type(4)));
typedef unsigned u4 __attribute__((ext_vector_type(4)));
typedef unsigned u8v __attribute__((ext_vector_type(8)));
typedef __attribute__((address_space(1))) unsigned gu32;
typedef __attribute__((address_space(4))) const unsigned cu32;
typedef __attribute__((address_space(3))) char lchar;

constexpr int NR = 8;
constexpr int KD = 2048, ND = 2048;
constexpr int MS = 256;
constexpr int KS = 256;
constexpr int NWG = 256;
constexpr int NTHREADS = 576;
constexpr unsigned long long ORD_LO = 0x7621054376543210ull, ORD_HI = 0x6527413075264130ull;
constexpr unsigned long long L0_LO = 0x7652104376543210ull, L0_HI = 0x6241305775413026ull;
constexpr int LMIN = 3, LSPAN = 5;
constexpr int FMIN = 10, FSPAN = 10;
constexpr int GLAST = 20;
constexpr unsigned RESPERM = 0x64201573u;
constexpr int AUX_SYS = 17, AUX_WT = 16;

constexpr unsigned CNT_OFF = 0;
constexpr unsigned FLAG_OFF = 32768;
constexpr unsigned TMO_OFF = 40960;
constexpr unsigned XG_OFF = 1u << 20;
constexpr unsigned XG_BYTES = 1u << 20;
constexpr unsigned WS_NEEDED = XG_OFF + 3 * XG_BYTES;
constexpr unsigned long long DEADLINE_TICKS = 50000000ull;

struct Args {
  const float* x;
  const float* w;
  float* out;
  char* myws;
  int rank;
  int pad;
  char* wsrot[NR];
};
static_assert(offsetof(Args, x) == 0 && offsetof(Args, w) == 8 && offsetof(Args, out) == 16 && offsetof(Args, myws) == 24 &&
              offsetof(Args, rank) == 32 && offsetof(Args, wsrot) == 0x28, "the kernel reads these by offset");

#define INLINE __attribute__((always_inline))
#define SCHED() __builtin_amdgcn_sched_barrier(0)
#define BAR() __builtin_amdgcn_s_barrier()

__device__ __forceinline__ unsigned cvt2(float a, float b) {
  f2 v = {a, b};
  return __builtin_bit_cast(unsigned, __builtin_convertvector(v, h2));
}
__device__ __forceinline__ unsigned dpp_hi(unsigned old, unsigned src) { return __builtin_amdgcn_update_dpp(old, src, 0x128, 0xf, 0xc, false); }
__device__ __forceinline__ unsigned dpp_lo(unsigned old, unsigned src) { return __builtin_amdgcn_update_dpp(old, src, 0x128, 0xf, 0x3, false); }
__device__ __forceinline__ bool any_zero(const u4* f) {
  unsigned mn = min(min(f[0].x, f[0].y), f[0].z);
  mn = min(min(mn, f[0].w), f[1].x); mn = min(min(mn, f[1].y), f[1].z);
  mn = min(min(mn, f[1].w), f[2].x); mn = min(min(mn, f[2].y), f[2].z);
  mn = min(min(mn, f[2].w), f[3].x); mn = min(min(mn, f[3].y), f[3].z);
  mn = min(mn, f[3].w);
  return __builtin_amdgcn_ballot_w64(mn == 0u) != 0ull;
}

__global__ __launch_bounds__(NTHREADS, 3) void a2a_gemm_relu(Args a) {
  __shared__ __attribute__((aligned(16))) char red[8 * 8192];
  u8v ka;
  unsigned rank;
  {
    const auto kp = __builtin_amdgcn_kernarg_segment_ptr();
    asm volatile("s_load_dwordx8 %0, %2, 0x0\n\ts_load_dword %1, %2, 0x20" : "=&s"(ka), "=&s"(rank) : "s"(kp));
    SCHED();
  }
  const int tid = threadIdx.x;
  const int lane = tid & 63;
  const int wave = __builtin_amdgcn_readfirstlane(tid >> 6);
  const unsigned b = blockIdx.x;
  const int c16 = lane & 15, q = lane >> 4;
  const unsigned xl = b & 7, j = b >> 3;
  const unsigned m = j & 3, ns = j >> 2;
  const unsigned n0 = 32 * ((RESPERM >> (4 * xl)) & 7) + 256 * ns;
  const bool comm = wave == 0;
  const unsigned p = (unsigned)(wave - 1) & 7;

  auto order = [&](bool l0, unsigned mm) INLINE -> unsigned {
    return (unsigned)(((mm & 2) ? (l0 ? L0_HI : ORD_HI) : (l0 ? L0_LO : ORD_LO)) >> (32 * (mm & 1)));
  };
  unsigned tab = order(xl == 0, m);
  const unsigned fi = 2 * ns + (m & 1), g = 16 * (m >> 1) + fi, um = g & 3;
  unsigned us = (order(false, um) >> (4 * (g >> 2))) & 7;
  unsigned dr7 = (xl - 1 + j) % 7;
  asm volatile("" : "+s"(dr7), "+s"(tab), "+s"(us));
  auto kstep = [&](int i) INLINE -> unsigned { return (tab >> (4 * i)) & 7; };
  const unsigned d = xl == 0 ? 7u : dr7;
  const int wvoff = ((8 * q + 2 * ((lane >> 3) & 1)) * ND + 4 * (lane & 7)) * 4;
  const int wbase = (KS * p * ND + n0) * 4;

  int xb0 = (int)((64 * um * KS + 32 * us) * 4), wb0 = wbase + (int)kstep(0) * 32 * ND * 4, wb1 = wbase + (int)kstep(1) * 32 * ND * 4;
  unsigned ap = xl == 0 ? 8 : m < 2 ? LMIN + (fi * LSPAN) / 16 : FMIN + (fi * FSPAN) / 16;
  asm volatile("" : "+s"(xb0), "+s"(wb0), "+s"(wb1), "+s"(ap));
  int fvoff = comm ? (c16 * KS + 8 * q) * 4 : wvoff;
  const int fb0 = comm ? xb0 : wb0, fb1 = comm ? xb0 + 32 * KS * 4 : wb1;
  const int fsa = comm ? 16 : ND * 4, fsb = comm ? 16 * KS * 4 : 4 * ND * 4;
  int fso[8];
#pragma unroll
  for (int n = 0; n < 8; ++n) fso[n] = (n < 4 ? fb0 : fb1) + (n & 1) * fsa + ((n >> 1) & 1) * fsb;
  unsigned frk = comm ? MS * KS * 4 : 0, fbytes = comm ? MS * KS * 4 : KD * ND * 4, dp1 = d + 1, cnto = CNT_OFF + b * 128;
  unsigned bpos = comm ? 99u : ap;
  asm volatile("" : "+v"(fvoff));
  asm volatile("" : "+s"(fso[0]), "+s"(fso[1]), "+s"(fso[2]), "+s"(fso[3]), "+s"(fso[4]), "+s"(fso[5]), "+s"(fso[6]), "+s"(fso[7]));
  asm volatile("" : "+s"(frk), "+s"(fbytes), "+s"(bpos), "+s"(dp1), "+s"(cnto));
  asm volatile("s_waitcnt lgkmcnt(0)" : "+s"(ka), "+s"(rank));
  const float* xp = (const float*)(((unsigned long long)ka[1] << 32) | ka[0]);
  const float* wp = (const float*)(((unsigned long long)ka[3] << 32) | ka[2]);
  float* outp = (float*)(((unsigned long long)ka[5] << 32) | ka[4]);
  char* myws = (char*)(((unsigned long long)ka[7] << 32) | ka[6]);
  f4 raw[32];
  unsigned cntv;
  {
    const char* fbase = (const char*)(comm ? xp : wp) + ((rank + dp1) & 7) * frk;
    asm volatile("" : "+s"(fbase));
    __amdgpu_buffer_rsrc_t fr = __builtin_amdgcn_make_buffer_rsrc((void*)fbase, 0, fbytes, 0x00020000);
    SCHED();
    if (!comm) BAR();
#pragma unroll
    for (int n = 0; n < 8; ++n) {
      if (n >= LMIN && n == (int)bpos) { BAR(); BAR(); }
      if (n == 4 && !comm) BAR();
      raw[n] = __builtin_bit_cast(f4, __builtin_amdgcn_raw_buffer_load_b128(fr, fvoff, fso[n], 0));
      if (n == 2) cntv = *(cu32*)(myws + cnto);
      SCHED();
    }
  }

  if (__builtin_expect(comm, 1)) {
    BAR();
    SCHED();
    __builtin_amdgcn_s_setprio(3);
    char* dws;
    asm volatile("s_load_dwordx2 %0, %1, %2 offset:0x28" : "=&s"(dws) : "s"(__builtin_amdgcn_kernarg_segment_ptr()), "s"((d & 7) * 8));
    SCHED();
    for (unsigned q4 = 4; q4 < ap; q4 += 4) BAR();
    SCHED();
    unsigned cv = cntv;
    asm volatile("s_waitcnt lgkmcnt(0)" : "+s"(dws), "+s"(cv));
    const unsigned call = cv + 1;
    __amdgpu_buffer_rsrc_t dr = __builtin_amdgcn_make_buffer_rsrc((void*)(dws + XG_OFF + (call % 3) * XG_BYTES), 0, XG_BYTES, 0x00020000);
    __amdgpu_buffer_rsrc_t zr = __builtin_amdgcn_make_buffer_rsrc((void*)(myws + XG_OFF + ((call + 2) % 3) * XG_BYTES), 0, XG_BYTES, 0x00020000);
    const int dbase = (int)(((rank * 4 + um) * 8 + us) * 4096);
    u4 h[4];
#pragma unroll
    for (int t = 0; t < 4; ++t) {
      const f4 lo = raw[2 * t], hi = raw[2 * t + 1];
      h[t] = u4{cvt2(lo.x, lo.y), cvt2(lo.z, lo.w), cvt2(hi.x, hi.y), cvt2(hi.z, hi.w)};
      asm volatile("" : "+v"(h[t]));
    }
    SCHED();
    BAR();
#pragma unroll
    for (int t = 0; t < 4; ++t) __builtin_amdgcn_raw_buffer_store_b128(h[t], dr, lane * 16 + t * 1024, dbase, AUX_SYS);
#pragma unroll
    for (int t = 0; t < 4; ++t) __builtin_amdgcn_raw_buffer_store_b128(u4{0u, 0u, 0u, 0u}, zr, lane * 16 + t * 1024, (int)(b * 4096), AUX_WT);
    SCHED();
    BAR();
    for (unsigned q4 = (ap + 3) & ~3u; q4 <= GLAST; q4 += 4) BAR();
    if (any_zero(h)) {
      asm volatile("s_waitcnt vmcnt(0)" ::: "memory");
      if (lane == 0)
        __hip_atomic_store((gu32*)(dws + FLAG_OFF + (rank * 4 + um) * 128 + us * 4), call, __ATOMIC_RELAXED, __HIP_MEMORY_SCOPE_SYSTEM);
    }
    asm volatile("s_endpgm" ::: "memory");
  }

  __amdgpu_buffer_rsrc_t wr = __builtin_amdgcn_make_buffer_rsrc((void*)wp, 0, KD * ND * 4, 0x00020000);
  auto wload = [&](unsigned s, int k) INLINE -> f4 {
    const int soff = wbase + (int)(s * 32 + (k & 1) + 4 * (k >> 1)) * ND * 4;
    return __builtin_bit_cast(f4, __builtin_amdgcn_raw_buffer_load_b128(wr, wvoff, soff, 0));
  };
  if (bpos == 8) { BAR(); BAR(); }
  BAR();
#pragma unroll
  for (int n = 8; n < 12; ++n) {
    if (n > 8 && (n < LMIN + LSPAN || n >= FMIN) && bpos == n) { BAR(); BAR(); }
    raw[n] = wload(kstep(2), n & 3);
    SCHED();
  }
  asm volatile("" : "+s"(cntv));
  const unsigned call = cntv + 1;
  __amdgpu_buffer_rsrc_t xr = __builtin_amdgcn_make_buffer_rsrc((void*)(myws + XG_OFF + (call % 3) * XG_BYTES + (p * 4 + m) * 32768), 0, 32768, 0x00020000);
  const int xvoff = lane * 16;

  unsigned seen = 0, flagged = 0, trusted = 0;
  bool timed_out = false;
  const unsigned long long t0 = __builtin_amdgcn_s_memrealtime();
  __amdgpu_buffer_rsrc_t pr = __builtin_amdgcn_make_buffer_rsrc((void*)myws, 0, WS_NEEDED, 0x00020000);
  int pxo = (int)(XG_OFF + (call % 3) * XG_BYTES + (p * 4 + m) * 32768 + 3072 + xl * 128 + 124), pfo = (int)(FLAG_OFF + (p * 4 + m) * 128);
  unsigned ptab = tab;
  asm volatile("" : "+s"(pxo), "+s"(pfo), "+s"(ptab));
  const int pks = (ptab >> (4 * (lane & 7))) & 7;
  int pvoff = lane < 8 ? pxo + pks * 4096 : lane < 16 ? pfo + pks * 4 : (int)TMO_OFF;
  asm volatile("" : "+v"(pvoff));
  constexpr int NOWHERE = 0x7fffffff;
  u4 xf[8][4];
  auto fetch = [&](int i) INLINE {
    if (__builtin_expect(!((seen >> i) & 1), 0)) {
      for (;;) {
        asm volatile("" ::: "memory");
        const unsigned pv = __builtin_amdgcn_raw_buffer_load_b32(pr, (lane < 8 && (lane < i || lane >= i + 4)) ? NOWHERE : pvoff, 0, AUX_SYS);
        flagged = ((unsigned)__builtin_amdgcn_ballot_w64((int)(pv - call) >= 0) >> 8) & 255u;
        seen = ((unsigned)__builtin_amdgcn_ballot_w64(pv != 0u) & 255u) | flagged;
        if ((((unsigned)__builtin_amdgcn_ballot_w64(pv == call) >> 16) & 1) || __builtin_amdgcn_s_memrealtime() - t0 > DEADLINE_TICKS) {
          __hip_atomic_store((gu32*)(myws + TMO_OFF), call, __ATOMIC_RELAXED, __HIP_MEMORY_SCOPE_AGENT);
          timed_out = true;
          seen = flagged = trusted = 255u;
        }
        if ((seen >> i) & 1) break;
        if (i < 7) __builtin_amdgcn_s_sleep(12); else __builtin_amdgcn_s_sleep(1);
      }
    }
    trusted |= flagged & (1u << i);
    asm volatile("" ::: "memory");
    SCHED();
#pragma unroll
    for (int mt = 0; mt < 4; ++mt) {
      xf[i][mt] = __builtin_amdgcn_raw_buffer_load_b128(xr, xvoff + mt * 1024, (int)(kstep(i) * 4096), AUX_SYS);
      SCHED();
    }
  };
  unsigned first = 0;

  h8 wf[8][2];
#pragma unroll
  for (int i = 0; i < 8; ++i) {
    u4 f0, f1;
#pragma unroll
    for (int h = 0; h < 2; ++h) {
      const f4 r0 = raw[4 * i + 2 * h], r1 = raw[4 * i + 2 * h + 1];
      const unsigned p0 = cvt2(r0.x, r1.x), p1 = cvt2(r0.y, r1.y), p2 = cvt2(r0.z, r1.z), p3 = cvt2(r0.w, r1.w);
      f0[2 * h] = dpp_hi(p0, p2); f0[2 * h + 1] = dpp_lo(p2, p0);
      f1[2 * h] = dpp_hi(p1, p3); f1[2 * h + 1] = dpp_lo(p3, p1);
    }
    wf[i][0] = __builtin_bit_cast(h8, f0);
    wf[i][1] = __builtin_bit_cast(h8, f1);
    asm volatile("" : "+v"(wf[i][0]), "+v"(wf[i][1]));
    if (i == 3) { first = __builtin_amdgcn_raw_buffer_load_b32(pr, lane >= 3 ? NOWHERE : pvoff, 0, AUX_SYS); SCHED(); }
    if (i == 5) {
      const unsigned mk = (unsigned)__builtin_amdgcn_ballot_w64(first != 0u);
      seen = mk == 7u ? 255u : mk;
    }
    if (i < 5) {
#pragma unroll
      for (int k = 0; k < 4; ++k) {
        const int n = 12 + 4 * i + k;
        if (n >= FMIN && n < FMIN + FSPAN && bpos == n) { BAR(); BAR(); }
        if (k == 0 && n <= GLAST) BAR();
        raw[n] = wload(kstep(3 + i), k);
        SCHED();
      }
    } else {
      fetch(i - 5);
    }
    SCHED();
  }

  f4 acc[4][2];
#pragma unroll
  for (int mt = 0; mt < 4; ++mt) { acc[mt][0] = (f4)0.f; acc[mt][1] = (f4)0.f; }
  int wa[4], ra, ovoff, osoff;
  auto put2 = [&](int mt) INLINE {
#pragma unroll
    for (int e = 0; e < 4; ++e) {
      *(__attribute__((address_space(3))) f2*)((lchar*)red + wa[e] + mt * 2048) = f2{acc[mt][0][e], acc[mt][1][e]};
      *(__attribute__((address_space(3))) f2*)((lchar*)red + wa[e] + mt * 2048 + 2048) = f2{acc[mt + 1][0][e], acc[mt + 1][1][e]};
    }
  };
#pragma unroll
  for (int i = 0; i < 8; ++i) {
    if (__builtin_expect(any_zero(xf[i]) && !((trusted >> i) & 1), 0)) {
      do {
        seen &= ~(1u << i);
#pragma unroll
        for (int e = i; e < i + 3 && e < 8; ++e) fetch(e);
      } while (any_zero(xf[i]) && !((trusted >> i) & 1));
    }
    if (i + 3 < 8) fetch(i + 3);
    if (i == 7 && __builtin_expect(timed_out, 0)) {
#pragma unroll
      for (int mt = 0; mt < 4; ++mt) { acc[mt][0] = (f4)__builtin_nanf(""); acc[mt][1] = acc[mt][0]; }
    }
#pragma unroll
    for (int mt = 0; mt < 4; ++mt) {
      acc[mt][0] = __builtin_amdgcn_mfma_f32_16x16x32_f16(wf[i][0], __builtin_bit_cast(h8, xf[i][mt]), acc[mt][0], 0, 0, 0);
      acc[mt][1] = __builtin_amdgcn_mfma_f32_16x16x32_f16(wf[i][1], __builtin_bit_cast(h8, xf[i][mt]), acc[mt][1], 0, 0, 0);
    }
    if (i == 7) {
      put2(0);
      __builtin_amdgcn_sched_group_barrier(0x8, 5, 0);
#pragma unroll
      for (int k = 0; k < 3; ++k) {
        __builtin_amdgcn_sched_group_barrier(0x2, 4, 0);
        __builtin_amdgcn_sched_group_barrier(0x200, 1, 0);
        __builtin_amdgcn_sched_group_barrier(0x8, 1, 0);
      }
      SCHED();
      put2(2);
    }
    if (i == 5) {
      const int wb = (int)p * 8192 + c16 * 128 + 8 * ((q >> 1) ^ (c16 >> 3));
#pragma unroll
      for (int e = 0; e < 4; ++e) { wa[e] = wb + (((4 * (q & 1) + e) ^ (c16 & 7)) * 16); asm volatile("" : "+v"(wa[e])); }
      const int row = 8 * (int)p + (lane >> 3), seg = lane & 7;
      ra = row * 128 + ((seg ^ (row & 7)) * 16);
      ovoff = (row * ND + 4 * seg) * 4;
      osoff = (int)((64 * m * ND + n0) * 4);
      asm volatile("" : "+v"(ra), "+v"(ovoff));
      asm volatile("" : "+s"(osoff));
    }
    SCHED();
  }
  __amdgpu_buffer_rsrc_t orr = __builtin_amdgcn_make_buffer_rsrc((void*)outp, 0, MS * ND * 4, 0x00020000);
  __syncthreads();
  {
    f4 v[8];
#pragma unroll
    for (int w = 0; w < 8; ++w) v[w] = *(const __attribute__((address_space(3))) f4*)((lchar*)red + ra + w * 8192);
    SCHED();
    f4 sum = ((v[0] + v[1]) + (v[2] + v[3])) + ((v[4] + v[5]) + (v[6] + v[7]));
    if (p & 1) sum = sum.zwxy;
    f4 o;
#pragma unroll
    for (int e = 0; e < 4; ++e) o[e] = __builtin_elementwise_maximum(sum[e], 0.0f);
    __builtin_amdgcn_raw_buffer_store_b128(__builtin_bit_cast(u4, o), orr, ovoff, osoff, AUX_WT);
  }
  if (tid == 64) __hip_atomic_store((gu32*)(myws + CNT_OFF + b * 128), call, __ATOMIC_RELAXED, __HIP_MEMORY_SCOPE_AGENT);
}

extern "C" void kernel_launch(int rank, int nranks, ncclComm_t comm, hipStream_t stream,
                              void* const* d_in, const int* in_sizes, int n_in,
                              void* d_out, int out_size,
                              const int* in_dtypes, int out_dtype,
                              void* const* d_out_all,
                              void* const* d_ws_all, size_t ws_bytes) {
  if (nranks != NR || (unsigned)rank >= (unsigned)NR || n_in < 2 || ws_bytes < WS_NEEDED ||
      in_sizes[0] != KD * KS || in_sizes[1] != KD * ND || out_size != MS * ND) {
    fprintf(stderr, "kernel_launch: unexpected shapes\n");
    return;
  }
  Args a;
  a.x = (const float*)d_in[0];
  a.w = (const float*)d_in[1];
  a.out = (float*)d_out;
  a.myws = (char*)d_ws_all[rank];
  a.rank = rank;
  a.pad = 0;
  for (int i = 0; i < NR; ++i) a.wsrot[i] = (char*)d_ws_all[(rank + 1 + i) & 7];
  hipLaunchKernelGGL(a2a_gemm_relu, dim3(NWG), dim3(NTHREADS), 0, stream, a);
  HIP_CHECK(hipGetLastError());
}
```
